```python
import numpy as np

kernel_name = "gpu_gemm_ar_m4096_k4096_n4096_e4m3e5m2_dqrelu"


ABSMAX_RTOL = 0.05

COMPUTE_DTYPE = "float32"
EPILOGUE = "dequant_relu"
IN_QUANT = None
OUT_QUANT = None
IN_FP8 = "e4m3e5m2"
IN_DTYPES = {"x": "e4m3", "w_mat": "e5m2",
             "scale_x": "float32", "scale_w": "float32"}
WORLD_SIZE = 8

def _snap_fp8(x, mant_bits, max_abs, min_normal):
    a = np.abs(x.astype(np.float64))
    _m, _e = np.frexp(a)
    q = np.where(a >= min_normal, np.ldexp(1.0, _e - 1 - mant_bits),
                 min_normal * 2.0 ** -mant_bits)
    snapped = np.minimum(np.round(a / q) * q, max_abs)
    return (np.sign(x) * snapped).astype(np.float32)

def setup_inputs():

    x = _snap_fp8(np.random.randn(4096, 4096).astype(np.float32), 3, 448.0, 2.0 ** -6)
    w_mat = _snap_fp8(np.random.randn(4096, 4096).astype(np.float32), 2, 57344.0, 2.0 ** -14)
    scale_x = (0.02 * (1.0 + np.random.rand(1))).astype(np.float32)
    scale_w = (0.02 * (1.0 + np.random.rand(1))).astype(np.float32)
    return {"x": x, "w_mat": w_mat, "scale_x": scale_x, "scale_w": scale_w}

def reference(x, w_mat, scale_x, scale_w):
    acc = (x @ w_mat).astype(np.float32)
    y = (acc * (scale_x * scale_w)).astype(np.float32)
    return np.maximum(y, 0.0).astype(np.float32)
```

```cpp
#include <hip/hip_runtime.h>
#include <rccl/rccl.h>
#include <cstdio>
#include <cstdint>

#define HIP_CHECK(expr) do { hipError_t _e = (expr); if (_e != hipSuccess) { \
    fprintf(stderr, "HIP error %s at %s:%d\n", hipGetErrorString(_e), __FILE__, __LINE__); \
    return; } } while (0)

typedef unsigned int u32;
typedef unsigned long long u64;
typedef __attribute__((ext_vector_type(2))) u32 u32x2;
typedef __attribute__((ext_vector_type(4))) u32 u32x4;
typedef __attribute__((ext_vector_type(8))) int i32x8;
typedef __attribute__((ext_vector_type(4))) float f32x4;
typedef __attribute__((ext_vector_type(16))) float f32x16;
#define LAS __attribute__((address_space(3)))
#define GAS __attribute__((address_space(1)))

constexpr int NR = 8;
constexpr int MM = 4096, NN = 4096, KS = 512;
constexpr u32 PIECE = 16384;
constexpr u32 OPB = 2u << 20;
constexpr u32 SRCB = 4u << 20;
constexpr u32 BUFB = 32u << 20;
constexpr u32 WS_FLAGS = 0;
constexpr u32 FLAGB = 8u * 2u * 128u * 64u;
constexpr u32 WS_CTR = 256u << 10;
constexpr u32 WS_TMO = 320u << 10;
constexpr u32 WS_DBG = 512u << 10;
constexpr u32 WS_DATA = 1u << 20;
constexpr size_t WS_NEED = (size_t)WS_DATA + 2 * (size_t)BUFB;
constexpr u64 DEADLINE = 100000000ull;

constexpr u32 NSLOT = 4;
constexpr u32 L_X = 0, L_W = NSLOT * PIECE, L_CTRL = 2 * NSLOT * PIECE;
constexpr u32 L_FULL = L_CTRL;
constexpr u32 L_TAIL = L_CTRL + 16;
constexpr u32 L_ABORT = L_CTRL + 20;
constexpr u32 L_CALL = L_CTRL + 24;
constexpr u32 L_DONE = L_CTRL + 64;
constexpr u32 L_QUEUE = L_CTRL + 128;
constexpr u32 L_BYTES = L_CTRL + 128 + 256;

constexpr int NBATCH = 32;
struct Params {
    const unsigned char* x;
    const unsigned char* w;
    const float* sx;
    const float* sw;
    float* out;
    unsigned char* ws[NR];
    int rank;
    int pad;
    u64 pieces[NBATCH + 1];
};
__host__ __device__ inline u32 mode_of(u64 pcs) { return ((u32)(pcs >> 7) & 1u) | ((u32)(pcs >> 14) & 2u); }

template <int N> __device__ __forceinline__ void waitv() { asm volatile("s_waitcnt vmcnt(%0)" ::"i"(N) : "memory"); }
__device__ __forceinline__ void waitl() { asm volatile("s_waitcnt lgkmcnt(0)" ::: "memory"); }
__device__ __forceinline__ u32 rfl(u32 v) { return __builtin_amdgcn_readfirstlane(v); }
__device__ __forceinline__ u64 rt() { return __builtin_amdgcn_s_memrealtime(); }

__device__ __forceinline__ u32 lds_rd(u32 addr) {
    u32 v;
    asm volatile("ds_read_b32 %0, %1\n\ts_waitcnt lgkmcnt(0)" : "=v"(v) : "v"(addr) : "memory");
    return v;
}
__device__ __forceinline__ u32x2 lds_rd2(u32 addr) {
    u32x2 v;
    asm volatile("ds_read_b64 %0, %1\n\ts_waitcnt lgkmcnt(0)" : "=v"(v) : "v"(addr) : "memory");
    return v;
}
__device__ __forceinline__ void lds_wr(u32 addr, u32 v) {
    asm volatile("ds_write_b32 %0, %1" ::"v"(addr), "v"(v) : "memory");
}

#define STAMP(slot) do { } while (0)

__global__ void __launch_bounds__(768) gemm_push_kernel(const Params p) {
    __shared__ __attribute__((aligned(16))) unsigned char smem[L_BYTES];
    const u32 lbase = (u32)(uintptr_t)(LAS unsigned char*)smem;
    const int wave = rfl(threadIdx.x >> 6);
    const int lane = threadIdx.x & 63;
    const u32 b = blockIdx.x;
    const u32 xcd = b & 7, idx = b >> 3;
    const u32 ti = (xcd >> 1) + 4 * (idx >> 3);
    const u32 tj = (xcd & 1) * 8 + (idx & 7);
    unsigned char* myws = p.ws[p.rank];

    STAMP(0);
    if (wave == 8) {
        const u32 dofs = idx & 7;
        const u32 m = (idx >> 3) * 8 + xcd;
        const u32 dest = ((u32)p.rank + dofs) & 7;
        __amdgpu_buffer_rsrc_t rx = __builtin_amdgcn_make_buffer_rsrc((void*)p.x, 0, (int)OPB, 0x00020000);
        __amdgpu_buffer_rsrc_t rw = __builtin_amdgcn_make_buffer_rsrc((void*)p.w, 0, (int)OPB, 0x00020000);
        const u32 vx = (u32)(lane & 31) * 512u + (u32)(lane >> 5) * 16u;
        const u32 vw = (u32)lane * 4u;
        const u32 vdx = (u32)lane * 16u;
        const u32 vdw = (u32)(lane >> 5) * 8192u + (u32)(lane & 31) * 16u;

        const u32 OOB = 0x40000000u;
        auto piece_of = [&](u64 pcs, u32 mode, u32 t, u32& kib) -> u32 {
            u32 pi;
            if (mode <= 1) { pi = m >> 2; kib = 4 * (m & 3) + t; }
            else if (mode == 2) { pi = 4 * (t >> 1) + (m >> 3); kib = 2 * (m & 7) + (t & 1); }
            else { pi = 2 * t + (m >> 4); kib = m & 15; }
            return (u32)(pcs >> (8 * pi)) & 127u;
        };
        u32x4 o[4];
        u32 vo[4], fl[4];
        auto load_w = [&](u64 pcs, u32 (&cur)[16]) {
            u32 kib; const u32 id = piece_of(pcs, 0, 0, kib);
            const u32 so = ((id >> 4) * 64u + (m & 3) * 16u) * 4096u + (id & 15) * 256u;
#pragma unroll
            for (int k = 0; k < 16; ++k) cur[k] = __builtin_amdgcn_raw_buffer_load_b32(rw, vw, so + k * 4096u, 0);
        };
        auto conv_w = [&](u64 pcs, const u32 (&cur)[16]) {
#pragma unroll
            for (int t = 0; t < 4; ++t) {
                const u32 d0 = cur[4 * t], d1 = cur[4 * t + 1], d2 = cur[4 * t + 2], d3 = cur[4 * t + 3];
                const u32 lo01 = __builtin_amdgcn_perm(d1, d0, 0x05010400u), hi01 = __builtin_amdgcn_perm(d1, d0, 0x07030602u);
                const u32 lo23 = __builtin_amdgcn_perm(d3, d2, 0x05010400u), hi23 = __builtin_amdgcn_perm(d3, d2, 0x07030602u);
                o[0][t] = __builtin_amdgcn_perm(lo23, lo01, 0x05040100u);
                o[1][t] = __builtin_amdgcn_perm(lo23, lo01, 0x07060302u);
                o[2][t] = __builtin_amdgcn_perm(hi23, hi01, 0x05040100u);
                o[3][t] = __builtin_amdgcn_perm(hi23, hi01, 0x07060302u);
            }
            u32 kib; const u32 id = piece_of(pcs, 0, 0, kib);
#pragma unroll
            for (int t = 0; t < 4; ++t) { vo[t] = vdw + OPB + id * PIECE + t * 2048u + (m & 3) * 512u; fl[t] = OOB; }
            fl[3] = lane < 4 ? (128u + id) * 64u + (m & 3) * 16u + lane * 4u : OOB;
        };
        auto load_x = [&](u64 pcs, u32x4 (&cx)[4]) {
            const u32 mode = mode_of(pcs);
#pragma unroll
            for (int t = 0; t < 4; ++t) {
                u32 kib; const u32 id = piece_of(pcs, mode, t, kib);
                const u32 so = ((id & 15) * 256u + (kib >> 1) * 32u) * 512u + (id >> 4) * 64u + (kib & 1) * 32u;
                cx[t] = __builtin_amdgcn_raw_buffer_load_b128(rx, vx, so, 0);
            }
        };
        auto conv_x = [&](u64 pcs, const u32x4 (&cx)[4]) {
            const u32 mode = mode_of(pcs);
            const u32 per = mode == 1 ? 4u : (mode == 2 ? 2u : 1u);
#pragma unroll
            for (int t = 0; t < 4; ++t) {
                o[t] = cx[t];
                u32 kib; const u32 id = piece_of(pcs, mode, t, kib);
                vo[t] = vdx + id * PIECE + kib * 1024u;
                const bool last = ((t + 1) & (per - 1)) == 0;
                fl[t] = (last && (u32)lane < per) ? id * 64u + (kib + 1 - per + lane) * 4u : OOB;
            }
        };
        {
            const u64 pcs = p.pieces[0];
            if (mode_of(pcs) == 0) { u32 cur[16]; load_w(pcs, cur); conv_w(pcs, cur); }
            else { u32x4 cx[4]; load_x(pcs, cx); conv_x(pcs, cx); }
        }
        asm volatile("" : "+v"(o[0]), "+v"(o[1]), "+v"(o[2]), "+v"(o[3]));
        __builtin_amdgcn_s_barrier();
        const u32 call = rfl(lds_rd(lbase + L_CALL));
        unsigned char* dws = p.ws[dest];
        __amdgpu_buffer_rsrc_t rd = __builtin_amdgcn_make_buffer_rsrc(dws + WS_DATA + (size_t)(call & 1) * BUFB + (size_t)p.rank * SRCB, 0, (int)SRCB, 0x00020000);
        __amdgpu_buffer_rsrc_t rf = __builtin_amdgcn_make_buffer_rsrc(dws + WS_FLAGS + (size_t)p.rank * (FLAGB / 8), 0, (int)(FLAGB / 8), 0x00020000);
        __builtin_amdgcn_s_setprio(2);
        u32 fl2 = OOB, fl3 = OOB;
#define PUSH_ST(t) do { __builtin_amdgcn_raw_buffer_store_b128(o[t], rd, vo[t], 0, 17); asm volatile("" ::: "memory"); } while (0)
#define PUSH_FL(f) do { __builtin_amdgcn_raw_buffer_store_b32(call, rf, f, 0, 17); asm volatile("" ::: "memory"); } while (0)
#define PUSH_BODY(NL, LOADS, CONV) do { \
            waitv<2>();              \
            PUSH_FL(fl2); \
            LOADS; \
            asm volatile("" ::: "memory"); \
            PUSH_ST(0); \
            waitv<NL + 2>();         \
            PUSH_FL(fl3); \
            PUSH_ST(1); \
            waitv<2>();              \
            PUSH_FL(fl[0]); \
            PUSH_ST(2); \
            waitv<2>();              \
            PUSH_FL(fl[1]); \
            PUSH_ST(3); \
            fl2 = fl[2]; fl3 = fl[3]; \
            CONV; \
            asm volatile("" : "+v"(o[0]), "+v"(o[1]), "+v"(o[2]), "+v"(o[3])); \
        } while (0)
#pragma nounroll
        for (int bb = 1; bb < NBATCH; ++bb) {
            const u64 pcs = p.pieces[bb];
            if (mode_of(pcs) == 0) { u32 cur[16]; PUSH_BODY(16, load_w(pcs, cur), conv_w(pcs, cur)); }
            else { u32x4 cx[4]; PUSH_BODY(4, load_x(pcs, cx), conv_x(pcs, cx)); }
        }
        PUSH_BODY(0, (void)0, (void)0);
        waitv<2>();
        PUSH_FL(fl2);
        waitv<1>();
        PUSH_FL(fl3);
        waitv<0>();
        STAMP(1);
    } else if (wave == 9) {
        GAS u32* ctr = (GAS u32*)(myws + WS_CTR + b * 128u);
        const u32 call = rfl(__hip_atomic_load(ctr, __ATOMIC_RELAXED, __HIP_MEMORY_SCOPE_AGENT)) + 1u;
        if (lane < 32) lds_wr(lbase + L_CTRL + lane * 4u, lane == (int)((L_CALL - L_CTRL) / 4) ? call : 0u);
        waitl();
        __builtin_amdgcn_s_barrier();
        if (lane == 0) __hip_atomic_store(ctr, call, __ATOMIC_RELAXED, __HIP_MEMORY_SCOPE_AGENT);
        __amdgpu_buffer_rsrc_t rf = __builtin_amdgcn_make_buffer_rsrc(myws + WS_FLAGS, 0, (int)FLAGB, 0x00020000);
        const u32 src = lane >> 3, op = (lane >> 2) & 1;
        const u32 vbase = (src * 256u + op * 128u + (op ? tj : ti)) * 64u + (lane & 3) * 16u;
        u32 nc = 0;
        u32 tail = 0;
        const u64 t0 = rt();
        __builtin_amdgcn_s_setprio(1);
        while (tail < 64) {
            u32x4 f;
            const u32 vo = vbase + (nc & 7) * 1024u;
            asm volatile("buffer_load_dwordx4 %0, %1, %2, 0 offen sc0 sc1\n\ts_waitcnt vmcnt(0)" : "=v"(f) : "v"(vo), "s"(rf) : "memory");
            bool ok = (int)(f.x - call) >= 0 && (int)(f.y - call) >= 0 && (int)(f.z - call) >= 0 && (int)(f.w - call) >= 0;
            if (rt() - t0 > DEADLINE) {
                ok = true;
                if (lane == 0) {
                    lds_wr(lbase + L_ABORT, 1u);
                    __hip_atomic_store((GAS u32*)(myws + WS_TMO), call, __ATOMIC_RELAXED, __HIP_MEMORY_SCOPE_AGENT);
                }
            }
            ok = ok && nc < 8;
            const u64 bal = __ballot(ok);
            const bool rdy = ((bal >> (src * 8)) & 0xffull) == 0xffull;
            const u64 lead = __ballot(rdy && (lane & 7) == 0);
            if (lead) {
                if (rdy && (lane & 7) == 0) {
                    const u32 pos = tail + __builtin_popcountll(lead & ((1ull << lane) - 1ull));
                    lds_wr(lbase + L_QUEUE + pos * 4u, src * SRCB + nc * 16u * PIECE);
                }
                tail += __builtin_popcountll(lead);
                waitl();
                if (lane == 0) lds_wr(lbase + L_TAIL, tail);
                if (rdy) nc += 1;
            } else {
                __builtin_amdgcn_s_sleep(4);
            }
        }
        waitl();
        STAMP(1);
    } else if (wave >= 10) {
        const u32 op = wave - 10;
        __builtin_amdgcn_s_barrier();
        const u32 call = rfl(lds_rd(lbase + L_CALL));
        __amdgpu_buffer_rsrc_t rs = __builtin_amdgcn_make_buffer_rsrc(myws + WS_DATA + (size_t)(call & 1) * BUFB, 0, (int)BUFB, 0x00020000);
        const u32 gofs = op * OPB + (op ? tj : ti) * PIECE;
        const u32 lring = lbase + (op ? L_W : L_X);
        const u32 vo = (u32)lane * 16u;
        u32 issued = 0, published = 0, tail = 0, freed = NSLOT;
        u32 spins = 0;
        const u64 t0 = rt();
        __builtin_amdgcn_s_setprio(1);
        while (published < 64) {
            bool can = issued < 64 && issued - published < 3;
            if (can && issued >= tail) { tail = rfl(lds_rd(lbase + L_TAIL)); can = issued < tail; }
            if (can && issued >= freed) {
                const u32 d = lds_rd(lbase + L_DONE + (lane & 7) * 4u);
                u32 mn = d;
                mn = min(mn, (u32)__builtin_amdgcn_update_dpp(0, (int)mn, 0xB1, 0xf, 0xf, false));
                mn = min(mn, (u32)__builtin_amdgcn_update_dpp(0, (int)mn, 0x4E, 0xf, 0xf, false));
                mn = min(mn, (u32)__builtin_amdgcn_update_dpp(0, (int)mn, 0x141, 0xf, 0xf, false));
                freed = rfl(mn) + NSLOT;
                can = issued < freed;
            }
            if (can) {
                const u32 so = rfl(lds_rd(lbase + L_QUEUE + issued * 4u)) + gofs;
                const u32 ld = lring + (issued & (NSLOT - 1)) * PIECE;
#pragma unroll
                for (int k = 0; k < 4; ++k) {
                    asm volatile("s_mov_b32 m0, %0\n\ts_nop 0\n\t"
                                 "buffer_load_dwordx4 %1, %2, %3 offen sc0 sc1 lds\n\t"
                                 "buffer_load_dwordx4 %1, %2, %3 offen offset:1024 sc0 sc1 lds\n\t"
                                 "buffer_load_dwordx4 %1, %2, %3 offen offset:2048 sc0 sc1 lds\n\t"
                                 "buffer_load_dwordx4 %1, %2, %3 offen offset:3072 sc0 sc1 lds"
                                 ::"s"(ld + k * 4096u), "v"(vo), "s"(rs), "s"(so + k * 4096u) : "memory");
                }
                issued += 1;
            } else if (issued > published) {
                const u32 o = issued - published;
                if (o == 1) waitv<0>(); else if (o == 2) waitv<16>(); else waitv<32>();
                published += 1;
                if (lane == 0) lds_wr(lbase + L_FULL + op * 4u, published);
            } else {
                __builtin_amdgcn_s_sleep(1);
                if ((++spins & 1023u) == 0 && rt() - t0 > 2 * DEADLINE) {
                    if (lane == 0) { lds_wr(lbase + L_ABORT, 1u); lds_wr(lbase + L_FULL + op * 4u, 64u); }
                    break;
                }
            }
        }
        waitl();
        STAMP(1);
    } else {
        const u32 wm = wave >> 1, wn = wave & 1;
        const float scale = __builtin_bit_cast(float, rfl(__builtin_bit_cast(u32, p.sx[0] * p.sw[0])));
        __builtin_amdgcn_s_barrier();
        const LAS unsigned char* xa = (const LAS unsigned char*)smem + L_X + wm * 4096u + lane * 16u;
        const LAS unsigned char* wa = (const LAS unsigned char*)smem + L_W + wn * 8192u + lane * 16u;
        f32x16 acc[2][4];
#pragma unroll
        for (int a = 0; a < 2; ++a)
#pragma unroll
            for (int e = 0; e < 4; ++e)
#pragma unroll
                for (int r = 0; r < 16; ++r) acc[a][e][r] = 0.f;
        u32 avail = 0, spins = 0;
        const u64 t0 = rt();
        auto frag = [&](const LAS unsigned char* base, u32 off) -> i32x8 {
            const u32x4 lo = *(const LAS u32x4*)(base + off);
            const u32x4 hi = *(const LAS u32x4*)(base + off + 1024);
            return i32x8{(int)lo.x, (int)lo.y, (int)lo.z, (int)lo.w, (int)hi.x, (int)hi.y, (int)hi.z, (int)hi.w};
        };
#define MFMA(a, e, A, B) acc[a][e] = __builtin_amdgcn_mfma_scale_f32_32x32x64_f8f6f4(A, B, acc[a][e], 0, 1, 0, 0, 0, 0)
        for (u32 n4 = 0; n4 < 64; n4 += NSLOT) {
#pragma unroll
            for (u32 u = 0; u < NSLOT; ++u) {
                const u32 n = n4 + u;
                while (avail <= n) {
                    const u32x2 f = lds_rd2(lbase + L_FULL);
                    avail = rfl(min(f.x, f.y));
                    if (avail <= n) {
                        __builtin_amdgcn_s_sleep(1);
                        if ((++spins & 1023u) == 0 && rt() - t0 > 3 * DEADLINE) { lds_wr(lbase + L_ABORT, 1u); avail = 64; }
                    }
                }
                asm volatile("" ::: "memory");
                const u32 so = u * PIECE;
                const i32x8 x0 = frag(xa, so), w0 = frag(wa, so), x1 = frag(xa, so + 2048);
                const i32x8 w1 = frag(wa, so + 2048);
                MFMA(0, 0, x0, w0);
                MFMA(1, 0, x1, w0);
                const i32x8 w2 = frag(wa, so + 4096);
                MFMA(0, 1, x0, w1);
                MFMA(1, 1, x1, w1);
                i32x8 w3 = frag(wa, so + 6144);
                MFMA(0, 2, x0, w2);
                MFMA(1, 2, x1, w2);
                waitl();
                lds_wr(lbase + L_DONE + wave * 4u, n + 1);
                asm volatile("" : "+v"(w3));
                MFMA(0, 3, x0, w3);
                MFMA(1, 3, x1, w3);
            }
        }
        STAMP(1);
        const bool bad = rfl(lds_rd(lbase + L_ABORT)) != 0;
        const float sc = bad ? __builtin_nanf("") : scale;
        __amdgpu_buffer_rsrc_t ro = __builtin_amdgcn_make_buffer_rsrc(p.out, 0, (int)(MM * (u32)NN * 4u), 0x00020000);
        const u32 sbase = (ti * 256u + wm * 64u) * (NN * 4u) + (tj * 256u + wn * 128u) * 4u;
        const u32 vo = sbase + (u32)(lane >> 5) * 4u * (NN * 4u) + (u32)(lane & 31) * 16u;
#pragma unroll
        for (int a = 0; a < 2; ++a)
#pragma unroll
            for (int r = 0; r < 16; ++r) {
                u32x4 v;
#pragma unroll
                for (int e = 0; e < 4; ++e) {
                    const float y = acc[a][e][r] * sc;
                    v[e] = __builtin_bit_cast(u32, y < 0.f ? 0.f : y);
                }
                const u32 row = a * 32 + (r & 3) + 8 * (r >> 2);
                __builtin_amdgcn_raw_buffer_store_b128(v, ro, vo + row * (NN * 4u), 0, 16);
            }
    }
}

static void build_schedule(Params& p) {
    int nb = 0;
    auto add = [&](int mode, int c, int blk0) {
        u64 v = 0;
        for (int k = 0; k < 8; ++k) v |= (u64)(c * 16 + blk0 + k) << (8 * k);
        p.pieces[nb++] = v | ((u64)(mode & 1) << 7) | ((u64)(mode & 2) << 14);
    };
    for (int c = 0; c < 8; ++c) {
        add(0, c, 0); add(0, c, 8);
        const int xm = c == 7 ? 3 : 1;
        add(xm, c, 0); add(xm, c, 8);
    }
    p.pieces[NBATCH] = 0;
}

extern "C" void kernel_launch(int rank, int nranks, ncclComm_t comm,
                              hipStream_t stream,
                              void* const* d_in, const int* in_sizes, int n_in,
                              void* d_out, int out_size,
                              const int* in_dtypes, int out_dtype,
                              void* const* d_out_all,
                              void* const* d_ws_all, size_t ws_bytes) {
    (void)comm; (void)in_dtypes; (void)out_dtype; (void)d_out_all;
    if (nranks != NR || (unsigned)rank >= (unsigned)NR || n_in < 4 || ws_bytes < WS_NEED || out_size != MM * NN ||
        in_sizes[0] != MM * KS || in_sizes[1] != KS * NN) {
        fprintf(stderr, "unexpected problem shape\n");
        return;
    }
    Params p;
    p.x = (const unsigned char*)d_in[0];
    p.w = (const unsigned char*)d_in[1];
    p.sx = (const float*)d_in[2];
    p.sw = (const float*)d_in[3];
    p.out = (float*)d_out;
    for (int r = 0; r < NR; ++r) p.ws[r] = (unsigned char*)d_ws_all[r];
    p.rank = rank;
    p.pad = 0;
    build_schedule(p);
    hipLaunchKernelGGL(gemm_push_kernel, dim3(256), dim3(768), 0, stream, p);
    HIP_CHECK(hipGetLastError());
}
```

```cpp
#include <hip/hip_runtime.h>
#include <rccl/rccl.h>
#include <cstdio>
#include <cstdint>
#include <cstdlib>

#define HIP_CHECK(expr) do { hipError_t _e = (expr); if (_e != hipSuccess) { \
    fprintf(stderr, "HIP error %s at %s:%d\n", hipGetErrorString(_e), __FILE__, __LINE__); \
    return; } } while (0)

typedef unsigned int u32;
typedef unsigned long long u64;
typedef __attribute__((ext_vector_type(2))) u32 u32x2;
typedef __attribute__((ext_vector_type(4))) u32 u32x4;
typedef __attribute__((ext_vector_type(8))) int i32x8;
typedef __attribute__((ext_vector_type(4))) float f32x4;
typedef __attribute__((ext_vector_type(16))) float f32x16;
#define LAS __attribute__((address_space(3)))
#define GAS __attribute__((address_space(1)))

constexpr int NR = 8;
constexpr int MM = 4096, NN = 4096, KS = 512;
constexpr u32 PIECE = 16384;
constexpr u32 OPB = 2u << 20;
constexpr u32 SRCB = 4u << 20;
constexpr u32 BUFB = 32u << 20;
constexpr u32 WS_FLAGS = 0;
constexpr u32 FLAGB = 2u * 16u * 64u * 16u;
constexpr u32 WS_CTR = 256u << 10;
constexpr u32 WS_LINK = 384u << 10;
constexpr u32 WS_TMO = 448u << 10;
constexpr u32 WS_DBG = 512u << 10;
constexpr u32 WS_DATA = 1u << 20;
constexpr size_t WS_NEED = (size_t)WS_DATA + 8 * (size_t)BUFB;
constexpr u64 DEADLINE = 100000000ull;

constexpr u32 NSLOT = 4;
constexpr u32 SLOTB = 2 * PIECE;
constexpr u32 L_CTRL = NSLOT * SLOTB;
constexpr u32 L_DONE = L_CTRL;
constexpr u32 L_ABORT = L_CTRL + 36;
constexpr u32 L_CALL = L_CTRL + 40;
constexpr u32 L_FULL = L_CTRL + 256;
constexpr u32 L_TRACE = L_CTRL + 512;
constexpr u32 L_BYTES = L_TRACE;

constexpr int NTHREADS = 640;
#define LDPOL ""
#define BK 8
constexpr u32 QW = 16 / BK;
constexpr u32 NBURST = 128 / BK;
constexpr u32 PPR = 32 / QW;
struct Params {
    const unsigned char* x;
    const unsigned char* w;
    const float* sx;
    const float* sw;
    float* out;
    unsigned char* ws[NR];
    int rank;
    u32 tau, tau_local;
    u32 lead;
    u32 dbg, nbufm, nsmooth, sublog;
    u64 sched[NR][PPR][NBURST / 8];
};

template <int N> __device__ __forceinline__ void waitv() { asm volatile("s_waitcnt vmcnt(%0)" ::"i"(N) : "memory"); }
__device__ __forceinline__ void waitl() { asm volatile("s_waitcnt lgkmcnt(0)" ::: "memory"); }
__device__ __forceinline__ u32 rfl(u32 v) { return __builtin_amdgcn_readfirstlane(v); }
__device__ __forceinline__ u64 rt() { return __builtin_amdgcn_s_memrealtime(); }

__device__ __forceinline__ u32 lds_rd(u32 addr) {
    u32 v;
    asm volatile("ds_read_b32 %0, %1\n\ts_waitcnt lgkmcnt(0)" : "=v"(v) : "v"(addr) : "memory");
    return v;
}
__device__ __forceinline__ u32x2 lds_rd2(u32 addr) {
    u32x2 v;
    asm volatile("ds_read_b64 %0, %1\n\ts_waitcnt lgkmcnt(0)" : "=v"(v) : "v"(addr) : "memory");
    return v;
}
__device__ __forceinline__ void lds_wr(u32 addr, u32 v) {
    asm volatile("ds_write_b32 %0, %1" ::"v"(addr), "v"(v) : "memory");
}

#define STAMP(slot) do { } while (0)
#define TRACE(kind, i) do { } while (0)
#define TRACEV(kind, i, v) do { } while (0)
#define TRACE_DUMP(kind) do { } while (0)

__global__ void __launch_bounds__(NTHREADS) gemm_push_kernel(const Params p) {
    __shared__ __attribute__((aligned(16))) unsigned char smem[L_BYTES];
    const u32 lbase = (u32)(uintptr_t)(LAS unsigned char*)smem;
    const int wave = rfl(threadIdx.x >> 6);
    const int lane = threadIdx.x & 63;
    const u32 b = blockIdx.x;
    const u32 xcd = b & 7, idx = b >> 3;
    const u32 ti = (xcd >> 1) + 4 * (idx >> 3);
    const u32 tj = (xcd & 1) * 8 + (idx & 7);
    unsigned char* myws = p.ws[p.rank];

    STAMP(0);
    if (wave == 8) {
        const u32 dofs = idx & 7;
        const u32 m = (idx >> 3) * 8 + xcd;
        const u32 dest = ((u32)p.rank + dofs) & 7;
        __amdgpu_buffer_rsrc_t rx = __builtin_amdgcn_make_buffer_rsrc((void*)p.x, 0, (int)OPB, 0x00020000);
        __amdgpu_buffer_rsrc_t rw = __builtin_amdgcn_make_buffer_rsrc((void*)p.w, 0, (int)OPB, 0x00020000);
        const u32 vx = (u32)(lane & 31) * 512u + (u32)(lane >> 5) * 16u;
        const u32 vw = (u32)lane * 4u;
        const u64 tstart = rt();
        const u32 pos = (m & 24u) | ((m + dofs) & 7u);
        const u32 grp = pos / QW, part = pos % QW;
        static_assert(BK == 8, "unit = 8 KiB");
        u32x4 o[8], o2[8];
        u32 vo, vs, vo2, vs2;
        u64 sch = p.sched[dofs][grp][0];
        auto load_w = [&](u32 id, u32 (&cur)[32]) {
            const u32 so = (((id >> 4) & 7) * 64u + part * 32u) * 4096u + (id & 15) * 256u;
#pragma unroll
            for (int k = 0; k < 32; ++k) cur[k] = __builtin_amdgcn_raw_buffer_load_b32(rw, vw, so + k * 4096u, 0);
        };
        auto conv_w = [&](u32 id, const u32 (&cur)[32]) {
#pragma unroll
            for (int t = 0; t < 8; ++t) {
                const u32 d0 = cur[4 * t], d1 = cur[4 * t + 1], d2 = cur[4 * t + 2], d3 = cur[4 * t + 3];
                const u32 lo01 = __builtin_amdgcn_perm(d1, d0, 0x05010400u), hi01 = __builtin_amdgcn_perm(d1, d0, 0x07030602u);
                const u32 lo23 = __builtin_amdgcn_perm(d3, d2, 0x05010400u), hi23 = __builtin_amdgcn_perm(d3, d2, 0x07030602u);
                const int g = t >> 2;
                o2[0 + g][t & 3] = __builtin_amdgcn_perm(lo23, lo01, 0x05040100u);
                o2[2 + g][t & 3] = __builtin_amdgcn_perm(lo23, lo01, 0x07060302u);
                o2[4 + g][t & 3] = __builtin_amdgcn_perm(hi23, hi01, 0x05040100u);
                o2[6 + g][t & 3] = __builtin_amdgcn_perm(hi23, hi01, 0x07060302u);
            }
            vo2 = id * PIECE + part * 2048u; vs2 = 4096u;
#pragma unroll
            for (int i = 0; i < 8; ++i) asm volatile("" : "+v"(o2[i]));
        };
        auto load_x = [&](u32 id, u32x4 (&cx)[8]) {
            const u32 so = ((id & 15) * 256u + part * 128u) * 512u + (id >> 4) * 64u;
#pragma unroll
            for (int i = 0; i < 8; ++i) cx[i] = __builtin_amdgcn_raw_buffer_load_b128(rx, vx, so + (i >> 1) * 32u * 512u + (i & 1) * 32u, 0);
        };
        auto conv_x = [&](u32 id, const u32x4 (&cx)[8]) {
            vo2 = id * PIECE + part * 8192u; vs2 = 2048u;
#pragma unroll
            for (int i = 0; i < 8; ++i) { o2[i] = cx[i]; asm volatile("" : "+v"(o2[i])); }
        };
        auto rotate = [&]() {
#pragma unroll
            for (int i = 0; i < 8; ++i) o[i] = o2[i];
            vo = vo2; vs = vs2;
        };
        u32 id = (u32)sch & 255u;
        if (id & 128) { u32 cur[32]; load_w(id, cur); conv_w(id, cur); }
        else { u32x4 cx[8]; load_x(id, cx); conv_x(id, cx); }
        rotate();
        __builtin_amdgcn_s_barrier();
        if (p.dbg & 32) return;
        const u32 call = rfl(lds_rd(lbase + L_CALL));
        unsigned char* dws = p.ws[dest];
        __amdgpu_buffer_rsrc_t rd = __builtin_amdgcn_make_buffer_rsrc(dws + WS_DATA + (size_t)(call & p.nbufm) * BUFB + (size_t)p.rank * SRCB, 0, (int)SRCB, 0x00020000);
        __amdgpu_buffer_rsrc_t rf = __builtin_amdgcn_make_buffer_rsrc(dws + WS_FLAGS, 0, (int)FLAGB, 0x00020000);
        __builtin_amdgcn_s_setprio(2);
        const u32 tau = dofs ? p.tau : p.tau_local;
        u32 round = (u32)tstart * 64u + p.lead;
        const u32 lane16 = (u32)lane * 16u, lanef = (u32)lane * FLAGB;
        const u32 smask = (1u << p.sublog) - 1u;
        u32 pflag = FLAGB;
#define FLAG_OF(id) (((((id) >> 7) * 16u + ((id) & 15)) * 64u + (((id) >> 4) & 7) * 8u + (u32)p.rank) * QW + part) * 4u
#define PACE(t) do { const u32 due_ = (t); while ((int)((u32)rt() * 64u - due_) < 0) __builtin_amdgcn_s_sleep(1); } while (0)
#define PST(i) do { __builtin_amdgcn_raw_buffer_store_b128(o[i], rd, lane16 + vo + ((i) >> 1) * vs + ((i) & 1) * 1024u, 0, 17); asm volatile("" ::: "memory"); } while (0)
#define PFL(f) do { __builtin_amdgcn_raw_buffer_store_b32(call, rf, (f) + lanef, 0, 17); asm volatile("" ::: "memory"); } while (0)
#define SST(i) do { if (((i) & smask) == 0) PACE(sbase + (i) * 4u * tau); PST(i); } while (0)
#define PUSH_SMOOTH(NL, LOADS, CONV) do { \
            const u32 sbase = round + ((pos * tau) >> (3 - p.sublog)); \
            SST(0); TRACE(0, 2 * k); SST(1); \
            LOADS; \
            asm volatile("" ::: "memory"); \
            SST(2); \
            waitv<3 + NL>();     \
            PFL(pflag); \
            SST(3); SST(4); SST(5); SST(6); \
            CONV; \
            SST(7); TRACE(0, 2 * k + 1); \
            pflag = FLAG_OF(id); \
        } while (0)
#define PUSH_BURST(NL, LOADS, CONV) do { \
            if (pflag != FLAGB) { waitv<0>(); PFL(pflag); pflag = FLAGB; } \
            PACE(round + pos * tau); \
            TRACE(0, 2 * k); \
            PST(0); PST(1); PST(2); PST(3); PST(4); PST(5); PST(6); PST(7); \
            LOADS; \
            waitv<NL>(); \
            TRACE(0, 2 * k + 1); \
            PFL(FLAG_OF(id)); \
            CONV; \
        } while (0)
#pragma nounroll
        for (u32 k = 0; k + 1 < NBURST; ++k) {
            sch >>= 8;
            if ((k & 7) == 7) sch = p.sched[dofs][grp][(k + 1) >> 3];
            const u32 nid = (u32)sch & 255u;
            if (k < p.nsmooth) {
                if (nid & 128) { u32 cur[32]; PUSH_SMOOTH(32, load_w(nid, cur), conv_w(nid, cur)); }
                else { u32x4 cx[8]; PUSH_SMOOTH(8, load_x(nid, cx), conv_x(nid, cx)); }
            } else {
                if (nid & 128) { u32 cur[32]; PUSH_BURST(32, load_w(nid, cur), conv_w(nid, cur)); }
                else { u32x4 cx[8]; PUSH_BURST(8, load_x(nid, cx), conv_x(nid, cx)); }
            }
            rotate();
            id = nid;
            round += 32u * tau;
        }
        { const u32 k = NBURST - 1; (void)k; PUSH_BURST(0, (void)0, (void)0); }
        waitv<0>();
        TRACE_DUMP(0);
        STAMP(1);
    } else if (wave == 9) {
        GAS u32* ctr = (GAS u32*)(myws + WS_CTR + b * 512u);
        const u32 call = rfl(__hip_atomic_load(ctr, __ATOMIC_RELAXED, __HIP_MEMORY_SCOPE_AGENT)) + 1u;
        lds_wr(lbase + L_CTRL + lane * 4u, lane == (int)((L_CALL - L_CTRL) / 4) ? call : 0u);
        lds_wr(lbase + L_FULL + lane * 4u, 0u);
        waitl();
        __builtin_amdgcn_s_barrier();
        if (lane == 0) __hip_atomic_store(ctr, call, __ATOMIC_RELAXED, __HIP_MEMORY_SCOPE_AGENT);
        ctr[64 + lane] = (u32)lane + 1u;
        waitv<0>();
        __amdgpu_buffer_rsrc_t rf = __builtin_amdgcn_make_buffer_rsrc(myws + WS_FLAGS, 0, (int)FLAGB, 0x00020000);
        __amdgpu_buffer_rsrc_t rtag = __builtin_amdgcn_make_buffer_rsrc(myws + WS_CTR + b * 512u + 256u, 0, 256, 0x00020000);
        __amdgpu_buffer_rsrc_t rs = __builtin_amdgcn_make_buffer_rsrc(myws + WS_DATA + (size_t)(call & p.nbufm) * BUFB, 0, (int)BUFB, 0x00020000);
        const u32 vfx = (ti * 64u + (u32)lane) * (QW * 4u), vfw = ((16u + tj) * 64u + (u32)lane) * (QW * 4u);
        const u32 vo = (u32)lane * 16u, vz = 0u;
        const u32 vctl = lbase + L_DONE + (u32)(lane & 7) * 4u;
        u64 seen = 0, pending = 0, tlast = 0; (void)tlast;
        u32 issued = 0, freed = NSLOT;
        const u32 t0 = (u32)rt();
        __builtin_amdgcn_s_setprio(1);
        while (issued < 64) {
            if (pending && issued >= freed) {
                u32 mn = lds_rd(vctl);
                mn = min(mn, (u32)__builtin_amdgcn_update_dpp(0, (int)mn, 0xB1, 0xf, 0xf, false));
                mn = min(mn, (u32)__builtin_amdgcn_update_dpp(0, (int)mn, 0x4E, 0xf, 0xf, false));
                mn = min(mn, (u32)__builtin_amdgcn_update_dpp(0, (int)mn, 0x141, 0xf, 0xf, false));
                freed = rfl(mn) + NSLOT;
            }
            const bool go = pending && issued < freed;
            const u64 p1 = pending & (pending - 1);
            const bool poll = seen != ~0ull && (go ? (p1 & (p1 - 1)) == 0 : p1 == 0);
            if (!go && !poll) {
                __builtin_amdgcn_s_sleep(1);
                if ((u32)rt() - t0 > (u32)(2 * DEADLINE)) { lds_wr(lbase + L_ABORT, 1u); break; }
                continue;
            }
            u32x2 fx, fw;
#define POLL_S "buffer_load_dwordx2 %[fx], %[vfx], %[rf], 0 offen sc0 sc1\n\tbuffer_load_dwordx2 %[fw], %[vfw], %[rf], 0 offen sc0 sc1\n\t"
#define DMA4_S(so) "buffer_load_dwordx4 %[vo], %[rs], " so " offen " LDPOL " lds\n\t" \
                   "buffer_load_dwordx4 %[vo], %[rs], " so " offen offset:1024 " LDPOL " lds\n\t" \
                   "buffer_load_dwordx4 %[vo], %[rs], " so " offen offset:2048 " LDPOL " lds\n\t" \
                   "buffer_load_dwordx4 %[vo], %[rs], " so " offen offset:3072 " LDPOL " lds\n\t" \
                   "s_add_u32 m0, m0, 0x1000\n\ts_add_u32 " so ", " so ", 0x1000\n\t"
#define STAGE_S "s_mov_b32 m0, %[ld]\n\ts_nop 0\n\t" \
                DMA4_S("%[sx]") DMA4_S("%[sx]") DMA4_S("%[sx]") DMA4_S("%[sx]") \
                DMA4_S("%[sw]") DMA4_S("%[sw]") DMA4_S("%[sw]") DMA4_S("%[sw]") \
                "s_mov_b32 m0, %[lf]\n\ts_nop 0\n\tbuffer_load_dword %[vz], %[rt], %[st] offen lds\n\t"
            if (go) {
                const u32 t = __builtin_ctzll(pending);
                pending = p1;
                const u32 so = (t & 7) * SRCB + (t >> 3) * 16u * PIECE;
                u32 sx = so + ti * PIECE, sw = so + OPB + tj * PIECE;
                const u32 ld = lbase + (issued & (NSLOT - 1)) * SLOTB;
                TRACE(6, issued);
                TRACEV(4, issued, (t & 7) * 8 + (t >> 3));
                if (poll)
                    asm volatile(POLL_S STAGE_S "s_waitcnt vmcnt(33)"
                                 : [fx] "=&v"(fx), [fw] "=&v"(fw), [sx] "+s"(sx), [sw] "+s"(sw)
                                 : [vfx] "v"(vfx), [vfw] "v"(vfw), [rf] "s"(rf), [vo] "v"(vo), [rs] "s"(rs), [ld] "s"(ld),
                                   [lf] "s"(lbase + L_FULL), [vz] "v"(vz), [rt] "s"(rtag), [st] "s"(issued * 4u) : "memory", "scc");
                else
                    asm volatile(STAGE_S
                                 : [sx] "+s"(sx), [sw] "+s"(sw)
                                 : [vo] "v"(vo), [rs] "s"(rs), [ld] "s"(ld),
                                   [lf] "s"(lbase + L_FULL), [vz] "v"(vz), [rt] "s"(rtag), [st] "s"(issued * 4u) : "memory", "scc");
                TRACE(7, issued);
                issued += 1;
                if (!poll) continue;
            } else {
                asm volatile(POLL_S "s_waitcnt vmcnt(0)"
                             : [fx] "=&v"(fx), [fw] "=&v"(fw) : [vfx] "v"(vfx), [vfw] "v"(vfw), [rf] "s"(rf) : "memory");
            }
            bool ok = (int)((fx.x - call) | (fx.y - call) | (fw.x - call) | (fw.y - call)) >= 0;
            if (p.dbg & 16) ok = true;
            u64 fresh = __ballot(ok) & ~seen;
            if (!fresh && !go) {
                if ((u32)rt() - t0 > (u32)DEADLINE) {
                    lds_wr(lbase + L_ABORT, 1u);
                    if (lane == 0) __hip_atomic_store((GAS u32*)(myws + WS_TMO), call, __ATOMIC_RELAXED, __HIP_MEMORY_SCOPE_AGENT);
                    fresh = ~seen;
                } else {
                    __builtin_amdgcn_s_sleep(1);
                }
            }
            seen |= fresh;
            pending |= fresh;
        }
        waitv<0>();
        waitl();
        TRACE_DUMP(1);
        TRACE_DUMP(4);
        TRACE_DUMP(6);
        TRACE_DUMP(7);
        STAMP(1);
    } else {
        const u32 wm = wave >> 1, wn = wave & 1;
        u32 scale_bits = rfl(__builtin_bit_cast(u32, p.sx[0] * p.sw[0]));
        asm volatile("" : "+s"(scale_bits));
        __builtin_amdgcn_s_barrier();
        u32 xa = lbase + wm * 4096u + (u32)lane * 16u;
        u32 wa = lbase + PIECE + (u32)(lane >> 5) * 1024u + wn * 512u + (u32)(lane & 31) * 16u;
        const u32 vfull = lbase + L_FULL, vdone = lbase + L_DONE + (u32)wave * 4u;
        f32x16 acc[2][4];
#pragma unroll
        for (int a = 0; a < 2; ++a)
#pragma unroll
            for (int e = 0; e < 4; ++e)
#pragma unroll
                for (int r = 0; r < 16; ++r) acc[a][e][r] = 0.f;
        u32 sn, sav, sspin, vt;
#define MF(acc, A, B) "v_mfma_f32_32x32x64_f8f6f4 " acc ", " A ", " B ", " acc " blgp:1\n\t"
#define RX0 "ds_read_b128 v[136:139], %8\n\tds_read_b128 v[140:143], %8 offset:1024\n\t"
#define RX1 "ds_read_b128 v[144:147], %8 offset:2048\n\tds_read_b128 v[148:151], %8 offset:3072\n\t"
#define RWA(o0, o1) "ds_read_b128 v[152:155], %9 offset:" #o0 "\n\tds_read_b128 v[156:159], %9 offset:" #o1 "\n\t"
#define RWB(o0, o1) "ds_read_b128 v[160:163], %9 offset:" #o0 "\n\tds_read_b128 v[164:167], %9 offset:" #o1 "\n\t"
#define X0 "v[136:143]"
#define X1 "v[144:151]"
#define WA "v[152:159]"
#define WB "v[160:167]"
        asm volatile(
            "s_mov_b32 %10, 0\n\t"
            "s_mov_b32 %12, 0\n"
            "1:\n\t"
            "ds_read_b32 %13, %14\n\t"
            "s_waitcnt lgkmcnt(0)\n\t"
            "v_readfirstlane_b32 %11, %13\n\t"
            "s_cmp_gt_u32 %11, %10\n\t"
            "s_cbranch_scc1 2f\n\t"
            "s_sleep 1\n\t"
            "s_add_u32 %12, %12, 1\n\t"
            "s_cmp_lt_u32 %12, 0x2000000\n\t"
            "s_cbranch_scc1 1b\n\t"
            "s_branch 5f\n"
            "2:\n\t"
            RWA(0, 2048) RX0 RX1 RWB(4096, 6144)
            "3:\n\t"
            "s_waitcnt lgkmcnt(4)\n\t"
            MF("%0", X0, WA)
            "s_waitcnt lgkmcnt(2)\n\t"
            MF("%4", X1, WA)
            RWA(8192, 10240)
            "ds_read_b32 %13, %14\n\t"
            "s_waitcnt lgkmcnt(3)\n\t"
            MF("%1", X0, WB)
            "s_add_u32 %10, %10, 1\n\t"
            MF("%5", X1, WB)
            RWB(12288, 14336)
            "v_add_u32 %8, 0x8000, %8\n\t"
            "v_add_u32 %9, 0x8000, %9\n\t"
            "v_and_b32 %8, 0x1ffff, %8\n\t"
            "v_and_b32 %9, 0x1ffff, %9\n\t"
            "s_waitcnt lgkmcnt(2)\n\t"
            MF("%2", X0, WA)
            "v_readfirstlane_b32 %11, %13\n\t"
            "v_mov_b32 %13, %10\n\t"
            MF("%6", X1, WA)
            "s_cmp_gt_u32 %11, %10\n\t"
            "s_cbranch_scc0 4f\n\t"
            RWA(0, 2048)
            "s_waitcnt lgkmcnt(2)\n\t"
            "ds_write_b32 %15, %13\n\t"
            MF("%3", X0, WB)
            RX0
            MF("%7", X1, WB)
            RX1 RWB(4096, 6144)
            "s_branch 3b\n"
            "4:\n\t"
            "s_waitcnt lgkmcnt(0)\n\t"
            "ds_write_b32 %15, %13\n\t"
            MF("%3", X0, WB)
            MF("%7", X1, WB)
            "s_cmp_lt_u32 %10, 64\n\t"
            "s_cbranch_scc1 1b\n"
            "5:\n\t"
            "s_waitcnt lgkmcnt(0)\n\t"
            "s_nop 15\n\t"
            "s_nop 7"
            : "+v"(acc[0][0]), "+v"(acc[0][1]), "+v"(acc[0][2]), "+v"(acc[0][3]),
              "+v"(acc[1][0]), "+v"(acc[1][1]), "+v"(acc[1][2]), "+v"(acc[1][3]),
              "+v"(xa), "+v"(wa), "=&s"(sn), "=&s"(sav), "=&s"(sspin), "=&v"(vt)
            : "v"(vfull), "v"(vdone)
            : "memory", "scc",
              "v136", "v137", "v138", "v139", "v140", "v141", "v142", "v143", "v144", "v145", "v146", "v147", "v148", "v149", "v150", "v151",
              "v152", "v153", "v154", "v155", "v156", "v157", "v158", "v159", "v160", "v161", "v162", "v163", "v164", "v165", "v166", "v167");
        const float scale = __builtin_bit_cast(float, scale_bits);
        STAMP(1);
        const bool bad = rfl(lds_rd(lbase + L_ABORT)) != 0 || sn < 64 || lbase != 0;
        const float sc = bad ? __builtin_nanf("") : scale;
        __amdgpu_buffer_rsrc_t ro = __builtin_amdgcn_make_buffer_rsrc(p.out, 0, (int)(MM * (u32)NN * 4u), 0x00020000);
        const u32 sbase = (ti * 256u + wm * 64u) * (NN * 4u) + (tj * 256u + wn * 128u) * 4u;
        const u32 vo = sbase + (u32)(lane >> 5) * 4u * (NN * 4u) + (u32)(lane & 31) * 16u;
#pragma unroll
        for (int a = 0; a < 2; ++a)
#pragma unroll
            for (int r = 0; r < 16; ++r) {
                u32x4 v;
#pragma unroll
                for (int e = 0; e < 4; ++e) {
                    const float y = acc[a][e][r] * sc;
                    v[e] = __builtin_bit_cast(u32, __builtin_elementwise_maximum(y, 0.f));
                }
                const u32 row = a * 32 + (r & 3) + 8 * (r >> 2);
                __builtin_amdgcn_raw_buffer_store_b128(v, ro, vo + row * (NN * 4u), 0, 16);
            }
    }
}

static void build_schedule(Params& p) {
    for (int d = 0; d < NR; ++d) {
        unsigned char order[256];
        int n = 0;
        for (int c = 0; c < 8; ++c) {
            if (c == 0) {
                for (int k = 0; k < 16; ++k) order[n++] = (unsigned char)(c * 16 + k);
                for (int k = 0; k < 16; ++k) order[n++] = (unsigned char)(128 + c * 16 + k);
            } else {
                for (int k = 0; k < 16; ++k) order[n++] = (unsigned char)(128 + c * 16 + k);
                for (int k = 0; k < 16; ++k) order[n++] = (unsigned char)(c * 16 + k);
            }
        }
        for (u32 g = 0; g < PPR; ++g)
            for (u32 j = 0; j < NBURST / 8; ++j) {
                u64 v = 0;
                for (u32 k = 0; k < 8; ++k) v |= (u64)order[(j * 8 + k) * PPR + g] << (8 * k);
                p.sched[d][g][j] = v;
            }
    }
}

extern "C" void kernel_launch(int rank, int nranks, ncclComm_t comm,
                              hipStream_t stream,
                              void* const* d_in, const int* in_sizes, int n_in,
                              void* d_out, int out_size,
                              const int* in_dtypes, int out_dtype,
                              void* const* d_out_all,
                              void* const* d_ws_all, size_t ws_bytes) {
    (void)comm; (void)in_dtypes; (void)out_dtype; (void)d_out_all;
    if (nranks != NR || (unsigned)rank >= (unsigned)NR || n_in < 4 || ws_bytes < WS_NEED || out_size != MM * NN ||
        in_sizes[0] != MM * KS || in_sizes[1] != KS * NN) {
        fprintf(stderr, "unexpected problem shape\n");
        return;
    }
    Params p;
    p.x = (const unsigned char*)d_in[0];
    p.w = (const unsigned char*)d_in[1];
    p.sx = (const float*)d_in[2];
    p.sw = (const float*)d_in[3];
    p.out = (float*)d_out;
    for (int r = 0; r < NR; ++r) p.ws[r] = (unsigned char*)d_ws_all[r];
    p.rank = rank;
    double gbs = 57.0, gbs_local = 120.0;
    p.lead = 6400; p.dbg = 0; p.nbufm = 1; p.nsmooth = 14; p.sublog = 1;
    p.tau = (u32)(BK * 1024.0 * 6.4 / gbs + 0.5);
    p.tau_local = (u32)(BK * 1024.0 * 6.4 / gbs_local + 0.5);
    build_schedule(p);
    hipLaunchKernelGGL(gemm_push_kernel, dim3(256), dim3(NTHREADS), 0, stream, p);
    HIP_CHECK(hipGetLastError());
}
```

```cpp
#include <hip/hip_runtime.h>
#include <rccl/rccl.h>
#include <cstdio>
#include <cstdint>
#include <cstdlib>

#define HIP_CHECK(expr) do { hipError_t _e = (expr); if (_e != hipSuccess) { \
    fprintf(stderr, "HIP error %s at %s:%d\n", hipGetErrorString(_e), __FILE__, __LINE__); \
    return; } } while (0)

typedef unsigned int u32;
typedef unsigned long long u64;
typedef __attribute__((ext_vector_type(2))) u32 u32x2;
typedef __attribute__((ext_vector_type(4))) u32 u32x4;
typedef __attribute__((ext_vector_type(8))) int i32x8;
typedef __attribute__((ext_vector_type(4))) float f32x4;
typedef __attribute__((ext_vector_type(16))) float f32x16;
#define LAS __attribute__((address_space(3)))
#define GAS __attribute__((address_space(1)))

constexpr int NR = 8;
constexpr int MM = 4096, NN = 4096, KS = 512;
constexpr u32 PIECE = 16384;
constexpr u32 OPB = 2u << 20;
constexpr u32 SRCB = 4u << 20;
constexpr u32 BUFB = 32u << 20;
constexpr u32 WS_FLAGS = 0;
constexpr u32 FLAGB = 2u * 16u * 64u * 8u;
constexpr u32 WS_TMO = 128u << 10;
constexpr u32 WS_T0 = 129u << 10;
constexpr u32 WS_CTR = 256u << 10;
constexpr u32 WS_DBG = 512u << 10;
constexpr u32 WS_DATA = 1u << 20;
constexpr size_t WS_NEED = (size_t)WS_DATA + 2 * (size_t)BUFB;
constexpr u32 DEADLINE = 100000000u;

constexpr u32 NSLOT = 4;
constexpr u32 SLOTB = 2 * PIECE;
constexpr u32 L_CTRL = NSLOT * SLOTB;
constexpr u32 L_DONE = L_CTRL;
constexpr u32 L_ABORT = L_CTRL + 36;
constexpr u32 L_CALL = L_CTRL + 40;
constexpr u32 L_PDONE = L_CTRL + 44;
constexpr u32 L_T0 = L_CTRL + 48;
constexpr u32 L_FULL = L_CTRL + 256;
constexpr u32 L_DESC = L_CTRL + 512;
constexpr u32 L_TRACE = L_CTRL + 1280;
constexpr u32 L_BYTES = L_TRACE;

constexpr int NTHREADS = 704;
#define PWIN 1
constexpr u32 NROUND = 16;
constexpr u32 RD_W = 128, RD_BURST = 64, RD_A = 16, RD_B = 32;
struct Params {
    const unsigned char* x;
    const unsigned char* w;
    const float* sx;
    const float* sw;
    float* out;
    unsigned char* ws[NR];
    u64 rounds[2][2];
    u64 split;
    int rank;
    u32 tau, tau_local;
    u32 lead;
    u32 taub, glog, pad, rotlog, epace, slowpoll, fastfrom, dbg, qlow, slack;
    u64 sched[NR][16][2];
};

template <int N> __device__ __forceinline__ void waitv() { asm volatile("s_waitcnt vmcnt(%0)" ::"i"(N) : "memory"); }
__device__ __forceinline__ void waitl() { asm volatile("s_waitcnt lgkmcnt(0)" ::: "memory"); }
__device__ __forceinline__ u32 rfl(u32 v) { return __builtin_amdgcn_readfirstlane(v); }
__device__ __forceinline__ u64 rt() { return __builtin_amdgcn_s_memrealtime(); }

__device__ __forceinline__ int lane_id() {
    int l;
    asm volatile("v_mbcnt_lo_u32_b32 %0, -1, 0\n\tv_mbcnt_hi_u32_b32 %0, -1, %0" : "=v"(l));
    return l;
}
__device__ __forceinline__ u32 lds_rd(u32 addr) {
    u32 v;
    asm volatile("ds_read_b32 %0, %1\n\ts_waitcnt lgkmcnt(0)" : "=v"(v) : "v"(addr) : "memory");
    return v;
}
__device__ __forceinline__ void lds_wr(u32 addr, u32 v) {
    asm volatile("ds_write_b32 %0, %1" ::"v"(addr), "v"(v) : "memory");
}

#define STAMP(slot) do { } while (0)
#define TRACE(kind, i) do { } while (0)
#define TRACEV(kind, i, v) do { } while (0)
#define TRACE_DUMP(kind) do { } while (0)

__global__ void __launch_bounds__(NTHREADS) gemm_push_kernel(const Params p) {
    __shared__ __attribute__((aligned(16))) unsigned char smem[L_BYTES];
    const u32 lbase = (u32)(uintptr_t)(LAS unsigned char*)smem;
    const int wave = rfl(threadIdx.x >> 6);
    const u32 b = blockIdx.x;
    const u32 xcd = b & 7, idx = b >> 3;
    const u32 ti = (xcd >> 1) + 4 * (idx >> 3);
    const u32 tj = (xcd & 1) * 8 + (idx & 7);
    unsigned char* myws = p.ws[0];

    STAMP(0);
    if (wave == 8) {
        const u64 tstart = rt();
        const int lane = lane_id();
        const u32 dofs = idx & 7;
        const u32 m = (idx >> 3) * 8 + xcd;
        __amdgpu_buffer_rsrc_t rx = __builtin_amdgcn_make_buffer_rsrc((void*)p.x, 0, (int)OPB, 0x00020000);
        __amdgpu_buffer_rsrc_t rw = __builtin_amdgcn_make_buffer_rsrc((void*)p.w, 0, (int)OPB, 0x00020000);
        const u32 vx = (u32)(lane & 31) * 512u + (u32)(lane >> 5) * 16u;
        const u32 vw = (u32)lane * 4u;
        const u32 pos = ((m & 24u) ^ p.pad) | ((m + dofs) & 7u);
        u32x4 o[8], o2[8];
        u32 vo, vs, vo2, vs2;
        u64 rds = p.rounds[dofs != 0][0];
        u64 sch = p.sched[dofs][pos >> 1][0];
        auto unit_of = [&](u32 rd) -> u32 {
            if (!(rd & (RD_A | RD_B))) return (((u32)sch & 255u) << 1) | (pos & 1);
            const u32 gm = (1u << p.rotlog) - 1u;
            const u32 blk = (pos & 15 & ~gm) | ((pos + ((dofs << p.rotlog) >> 3)) & gm);
            return (((rd & 7) + (pos >> 4)) << 5) | (blk << 1) | ((rd / RD_B) & 1);
        };
        auto load_w = [&](u32 u, u32 (&cur)[32]) {
            const u32 so = (((u >> 5) & 7) * 64u + (u & 1) * 32u) * 4096u + ((u >> 1) & 15) * 256u;
#pragma unroll
            for (int k = 0; k < 32; ++k) cur[k] = __builtin_amdgcn_raw_buffer_load_b32(rw, vw, so + k * 4096u, 0);
        };
        auto conv_w = [&](u32 u, const u32 (&cur)[32]) {
#pragma unroll
            for (int t = 0; t < 8; ++t) {
                const u32 d0 = cur[4 * t], d1 = cur[4 * t + 1], d2 = cur[4 * t + 2], d3 = cur[4 * t + 3];
                const u32 lo01 = __builtin_amdgcn_perm(d1, d0, 0x05010400u), hi01 = __builtin_amdgcn_perm(d1, d0, 0x07030602u);
                const u32 lo23 = __builtin_amdgcn_perm(d3, d2, 0x05010400u), hi23 = __builtin_amdgcn_perm(d3, d2, 0x07030602u);
                const int g = t >> 2;
                o2[0 + g][t & 3] = __builtin_amdgcn_perm(lo23, lo01, 0x05040100u);
                o2[2 + g][t & 3] = __builtin_amdgcn_perm(lo23, lo01, 0x07060302u);
                o2[4 + g][t & 3] = __builtin_amdgcn_perm(hi23, hi01, 0x05040100u);
                o2[6 + g][t & 3] = __builtin_amdgcn_perm(hi23, hi01, 0x07060302u);
            }
            vo2 = (u >> 1) * PIECE + (u & 1) * 2048u; vs2 = 4096u;
#pragma unroll
            for (int i = 0; i < 8; ++i) asm volatile("" : "+v"(o2[i]));
        };
        auto load_x = [&](u32 u, u32x4 (&cx)[8]) {
            const u32 so = (u & 31) * (128u * 512u) + (u >> 5) * 64u;
#pragma unroll
            for (int i = 0; i < 8; ++i) cx[i] = __builtin_amdgcn_raw_buffer_load_b128(rx, vx, so + (i >> 1) * 32u * 512u + (i & 1) * 32u, 0);
        };
        auto conv_x = [&](u32 u, const u32x4 (&cx)[8]) {
            vo2 = u * 8192u; vs2 = 2048u;
#pragma unroll
            for (int i = 0; i < 8; ++i) { o2[i] = cx[i]; asm volatile("" : "+v"(o2[i])); }
        };
        auto rotate = [&]() {
#pragma unroll
            for (int i = 0; i < 8; ++i) o[i] = o2[i];
            vo = vo2; vs = vs2;
        };
        u32 un = pos;
        { u32x4 cx[8]; load_x(un, cx); conv_x(un, cx); }
        rotate();
        __builtin_amdgcn_s_barrier();
        if (p.dbg & 32) __builtin_amdgcn_endpgm();
        const u32 call = rfl(lds_rd(lbase + L_CALL));
        unsigned char* dws = p.ws[dofs];
        __amdgpu_buffer_rsrc_t rd = __builtin_amdgcn_make_buffer_rsrc(dws + WS_DATA + (size_t)(call & 1) * BUFB + (size_t)p.rank * SRCB, 0, (int)SRCB, 0x00020000);
        __amdgpu_buffer_rsrc_t rf = __builtin_amdgcn_make_buffer_rsrc(dws + WS_FLAGS, 0, (int)FLAGB, 0x00020000);
        __builtin_amdgcn_s_setprio(3);
        const u32 tau = dofs ? p.tau : p.tau_local;
        u32 round = (u32)tstart * 64u + p.lead;
        const u32 lane16 = (u32)lane * 16u, lanef = (u32)lane * FLAGB;
        const u32 taub = dofs ? p.taub : p.tau_local;
        u32 pflag = FLAGB;
#define FLAG_OF(u) ((((((u) >> 8) * 16u + (((u) >> 1) & 15)) * 64u + (((u) >> 5) & 7) * 8u + (u32)p.rank) * 2u + ((u) & 1)) * 4u)
#define PACE(t) do { const u32 due_ = (t); while ((int)((u32)rt() * 64u - due_) < 0) __builtin_amdgcn_s_sleep(1); } while (0)
#define PST(i) do { __builtin_amdgcn_raw_buffer_store_b128(o[i], rd, lane16 + vo + ((i) >> 1) * vs + ((i) & 1) * 1024u, 0, 17); asm volatile("" ::: "memory"); } while (0)
#define PFL(f) do { __builtin_amdgcn_raw_buffer_store_b32(call, rf, (f) + lanef, 0, 17); asm volatile("" ::: "memory"); } while (0)
#define LATE(t) do { const int d_ = (int)((u32)rt() * 64u - (t)) - (int)p.slack; if (d_ > 0) { round += (u32)d_; sbase += (u32)d_; } } while (0)
#define PUSH_SMOOTH(NL, LOADS, CONV) do { \
            u32 sbase = round + ((pos * tau) >> 2); \
            PACE(sbase); waitv<PWIN>(); LATE(sbase); PST(0); TRACE(0, 2 * k); PST(1); \
            LOADS; \
            asm volatile("" ::: "memory"); \
            PACE(sbase + 8u * tau); waitv<PWIN + NL>(); LATE(sbase + 8u * tau); \
            PFL(pflag);          \
            PST(2); PST(3); \
            PACE(sbase + 16u * tau); waitv<PWIN>(); LATE(sbase + 16u * tau); PST(4); PST(5); \
            CONV; \
            PACE(sbase + 24u * tau); waitv<PWIN>(); LATE(sbase + 24u * tau); PST(6); PST(7); TRACE(0, 2 * k + 1); \
            pflag = FLAG_OF(un); \
            round += 32u * tau; \
        } while (0)
#define PUSH_BURST(NL, LOADS, CONV) do { \
            if (pflag != FLAGB) { waitv<0>(); PFL(pflag); pflag = FLAGB; } \
            const u32 gb = round + ((((pos >> p.glog) << (p.glog + 2)) + (pos & ((1u << p.glog) - 1u))) * taub >> 2); \
            const u32 gs = (taub << p.glog) >> 2; \
            PACE(gb); \
            TRACE(0, 2 * k); \
            PST(0); PST(1); PACE(gb + gs); PST(2); PST(3); PACE(gb + 2u * gs); PST(4); PST(5); PACE(gb + 3u * gs); PST(6); PST(7); \
            LOADS; \
            waitv<NL>(); \
            TRACE(0, 2 * k + 1); \
            PFL(FLAG_OF(un)); \
            CONV; \
            round += 32u * taub; \
        } while (0)
#pragma nounroll
        for (u32 k = 0; k + 1 < NROUND; ++k) {
            const bool burst = (u32)rds & RD_BURST;
            if (k == 2 && !(p.dbg & 512)) {
                const u32 tz = rfl(lds_rd(lbase + L_T0)), dl = (u32)tstart - tz;
                if (tz && dl < 300u) round -= dl * 64u;
            }
            rds >>= 8; sch >>= 8;
            if (k == 7) { rds = p.rounds[dofs != 0][1]; sch = p.sched[dofs][pos >> 1][1]; }
            const u32 nu = unit_of((u32)rds & 255u);
            if (!burst) {
                if (nu & 256) { u32 cur[32]; PUSH_SMOOTH(32, load_w(nu, cur), conv_w(nu, cur)); }
                else { u32x4 cx[8]; PUSH_SMOOTH(8, load_x(nu, cx), conv_x(nu, cx)); }
            } else {
                if (nu & 256) { u32 cur[32]; PUSH_BURST(32, load_w(nu, cur), conv_w(nu, cur)); }
                else { u32x4 cx[8]; PUSH_BURST(8, load_x(nu, cx), conv_x(nu, cx)); }
            }
            rotate();
            un = nu;
        }
        { const u32 k = NROUND - 1; (void)k; PUSH_BURST(0, (void)0, (void)0); }
        lds_wr(lbase + L_PDONE, 1u);
        waitv<0>();
        TRACE_DUMP(0);
        STAMP(1);
        __builtin_amdgcn_endpgm();
    } else if (wave == 9) {
        const u32 t0 = (u32)rt();
        const int lane = lane_id();
        GAS u32* ctr = (GAS u32*)(myws + WS_CTR + b * 1024u);
        const u32 call = rfl(__hip_atomic_load(ctr, __ATOMIC_RELAXED, __HIP_MEMORY_SCOPE_AGENT)) + 1u;
        lds_wr(lbase + L_CTRL + lane * 4u, lane == (int)((L_CALL - L_CTRL) / 4) ? call : 0u);
#pragma unroll
        for (u32 q = 256; q < 1280; q += 256) lds_wr(lbase + L_CTRL + q + lane * 4u, 0u);
        waitl();
        __builtin_amdgcn_s_barrier();
        if (lane == 0) __hip_atomic_store(ctr, call, __ATOMIC_RELAXED, __HIP_MEMORY_SCOPE_AGENT);
        {
            GAS u64* tw = (GAS u64*)(myws + WS_T0);
            if (b == 0 && lane == 0) __hip_atomic_store(tw, ((u64)call << 32) | t0, __ATOMIC_RELAXED, __HIP_MEMORY_SCOPE_AGENT);
            for (u32 q = 0; q < 6; ++q) {
                __builtin_amdgcn_s_sleep(16);
                const u64 v = __hip_atomic_load(tw, __ATOMIC_RELAXED, __HIP_MEMORY_SCOPE_AGENT);
                if (rfl((u32)(v >> 32)) == call) { lds_wr(lbase + L_T0, (u32)v | 1u); break; }
            }
        }
        __amdgpu_buffer_rsrc_t rf = __builtin_amdgcn_make_buffer_rsrc(myws + WS_FLAGS, 0, (int)FLAGB, 0x00020000);
        const u32 vfx = (ti * 64u + (u32)lane) * 8u, vfw = ((16u + tj) * 64u + (u32)lane) * 8u;
        const u32 task = (u32)(lane & 7) * SRCB + (u32)(lane >> 3) * 16u * PIECE;
        const u64 below = (1ull << lane) - 1ull;
        u64 seenA = 0, seenB = 0;
        u32 tail = 0;
        __builtin_amdgcn_s_setprio(2);
        while ((seenA & seenB) != ~0ull) {
            u32x2 fx, fw;
            asm volatile("buffer_load_dwordx2 %[fx], %[vfx], %[rf], 0 offen sc0 sc1\n\t"
                         "buffer_load_dwordx2 %[fw], %[vfw], %[rf], 0 offen sc0 sc1\n\t"
                         "s_waitcnt vmcnt(0)"
                         : [fx] "=&v"(fx), [fw] "=&v"(fw) : [vfx] "v"(vfx), [vfw] "v"(vfw), [rf] "s"(rf) : "memory");
            const u32 wok = (fw.x - call) | (fw.y - call);
            u64 rA = __ballot((int)(wok | (fx.x - call)) >= 0), rB = __ballot((int)(wok | (fx.y - call)) >= 0);
            if (p.dbg & 16) { rA = ~0ull; rB = (p.dbg & 64) && seenA == 0 ? ~p.split : ~0ull; }
            const u64 both = rA & rB;
            u64 fA = (both | (rA & p.split)) & ~seenA, fB = (both | (rB & p.split)) & ~seenB;
            const u32 el = (u32)rt() - t0;
            if (el > DEADLINE) {
                lds_wr(lbase + L_ABORT, 1u);
                if (lane == 0) __hip_atomic_store((GAS u32*)(myws + WS_TMO), call, __ATOMIC_RELAXED, __HIP_MEMORY_SCOPE_AGENT);
                fA = ~seenA; fB = ~seenB;
            }
            const u64 fr = fA | fB;
            if (fr) {
                const u64 fo = fB & ~fA;
                const u32 kind = (u32)((fA >> lane) & 1) | ((u32)((fB >> lane) & 1) << 1);
                const u32 at = tail + ((kind & 1) ? __builtin_popcountll(fA & below) : __builtin_popcountll(fA) + __builtin_popcountll(fo & below));
                if (kind) {
                    lds_wr(lbase + L_DESC + at * 4u, task | kind);
                    TRACE(1, at);
                    TRACEV(4, at, (lane & 7) * 8 + (lane >> 3) + kind * 64);
                }
                tail += __builtin_popcountll(fr);
                seenA |= fA; seenB |= fB;
            }
            if (el < p.fastfrom)
                for (u32 q = 0; q < p.slowpoll; ++q) __builtin_amdgcn_s_sleep(8);
            for (u32 q = 0; q < 4096 && tail > rfl(lds_rd(lbase + L_FULL)) + p.qlow; ++q) __builtin_amdgcn_s_sleep(2);
        }
        lds_wr(lbase + L_DESC + tail * 4u, 4u);
        STAMP(2);
        waitl();
        TRACE_DUMP(1);
        TRACE_DUMP(4);
        __builtin_amdgcn_endpgm();
    } else if (wave == 10) {
        const int lane = lane_id();
        GAS u32* tab = (GAS u32*)(myws + WS_CTR + b * 1024u + 512u);
        tab[lane] = (u32)lane + 1u;
        tab[64 + lane] = (u32)lane + 65u;
        __builtin_amdgcn_s_barrier();
        const u32 call = rfl(lds_rd(lbase + L_CALL));
        waitv<0>();
        __amdgpu_buffer_rsrc_t rtag = __builtin_amdgcn_make_buffer_rsrc((void*)tab, 0, 512, 0x00020000);
        __amdgpu_buffer_rsrc_t rs = __builtin_amdgcn_make_buffer_rsrc(myws + WS_DATA + (size_t)(call & 1) * BUFB, 0, (p.dbg & 1) ? 0 : (int)BUFB, 0x00020000);
        const u32 vo = (u32)lane * 16u, vz = 0u;
        const u32 vdn = lbase + L_DONE + (u32)(lane & 7) * 4u;
        u32 vq = lbase + L_DESC, vt, vd;
        u32 si, sq, sx, sw, sk, sl, sok, sp;
        __builtin_amdgcn_s_setprio(2);
#define DMA4_S(so) "buffer_load_dwordx4 %[vo], %[rs], " so " offen lds\n\t" \
                   "buffer_load_dwordx4 %[vo], %[rs], " so " offen offset:1024 lds\n\t" \
                   "buffer_load_dwordx4 %[vo], %[rs], " so " offen offset:2048 lds\n\t" \
                   "buffer_load_dwordx4 %[vo], %[rs], " so " offen offset:3072 lds\n\t" \
                   "s_add_u32 m0, m0, 0x1000\n\ts_add_u32 " so ", " so ", 0x1000\n\t"
        asm volatile(
            "s_mov_b32 %[si], 0\n\t"
            "s_mov_b32 %[sp], 0\n\t"
            "s_mov_b32 %[sl], %[lb]\n"
            "1:\n\t"
            "ds_read_b32 %[vt], %[vq]\n\t"
            "ds_read_b32 %[vd], %[vdn]\n\t"
            "s_waitcnt lgkmcnt(0)\n\t"
            "v_readfirstlane_b32 %[sq], %[vt]\n\t"
            "v_lshl_add_u32 %[vd], %[vd], 2, 16\n\t"
            "v_cmp_lt_u32 vcc, %[si], %[vd]\n\t"
            "s_cmp_eq_u32 %[sq], 4\n\t"
            "s_cbranch_scc1 9f\n\t"
            "s_cmp_eq_u64 vcc, -1\n\t"
            "s_cselect_b32 %[sok], %[sq], 0\n\t"
            "s_cmp_lg_u32 %[sok], 0\n\t"
            "s_cbranch_scc1 2f\n\t"
            "s_sleep 1\n\t"
            "s_add_u32 %[sp], %[sp], 1\n\t"
            "s_cmp_lt_u32 %[sp], 0x2000000\n\t"
            "s_cbranch_scc1 1b\n\t"
            "s_branch 9f\n"
            "2:\n\t"
            "s_mov_b32 m0, %[sl]\n\t"
            "s_and_b32 %[sk], %[sq], 3\n\t"
            "s_andn2_b32 %[sq], %[sq], 3\n\t"
            "s_add_u32 %[sx], %[sq], %[ox]\n\t"
            "s_add_u32 %[sw], %[sq], %[ow]\n\t"
            "s_bitcmp1_b32 %[sk], 0\n\ts_cbranch_scc0 3f\n\t"
            DMA4_S("%[sx]") DMA4_S("%[sx]")
            "s_branch 4f\n"
            "3:\n\ts_add_u32 m0, m0, 0x2000\n\ts_add_u32 %[sx], %[sx], 0x2000\n"
            "4:\n\ts_bitcmp1_b32 %[sk], 1\n\ts_cbranch_scc0 5f\n\t"
            DMA4_S("%[sx]") DMA4_S("%[sx]")
            "s_branch 6f\n"
            "5:\n\ts_add_u32 m0, m0, 0x2000\n\ts_nop 0\n"
            "6:\n\t"
            DMA4_S("%[sw]") DMA4_S("%[sw]")
            "ds_read_b32 %[vt], %[vq] offset:4\n\t"
            "ds_read_b32 %[vd], %[vdn]\n\t"
            "v_add_u32 %[vq], 4, %[vq]\n\t"
            "s_add_u32 %[sl], %[sl], 0x8000\n\t"
            "s_and_b32 %[sl], %[sl], 0x1ffff\n\t"
            DMA4_S("%[sw]")
            "s_waitcnt lgkmcnt(0)\n\t"
            "v_readfirstlane_b32 %[sq], %[vt]\n\t"
            "v_lshl_add_u32 %[vd], %[vd], 2, 12\n\t"
            "v_cmp_lt_u32 vcc, %[si], %[vd]\n\t"
            DMA4_S("%[sw]")
            "s_mov_b32 m0, %[lf]\n\t"
            "s_and_b32 %[sok], %[sq], 3\n\t"
            "buffer_load_dword %[vz], %[rt], %[si] offen lds\n\t"
            "s_add_u32 %[si], %[si], 4\n\t"
            "s_cmp_eq_u64 vcc, -1\n\t"
            "s_cselect_b32 %[sok], %[sok], 0\n\t"
            "s_cmp_lg_u32 %[sok], 0\n\t"
            "s_cbranch_scc1 2b\n\t"
            "s_branch 1b\n"
            "9:\n\t"
            "s_waitcnt vmcnt(0)"
            : [vq] "+v"(vq), [vt] "=&v"(vt), [vd] "=&v"(vd), [si] "=&s"(si), [sq] "=&s"(sq), [sx] "=&s"(sx), [sw] "=&s"(sw),
              [sk] "=&s"(sk), [sl] "=&s"(sl), [sok] "=&s"(sok), [sp] "=&s"(sp)
            : [vdn] "v"(vdn), [vo] "v"(vo), [vz] "v"(vz), [rs] "s"(rs), [rt] "s"(rtag), [lb] "s"(lbase), [lf] "s"(lbase + L_FULL),
              [ox] "s"(ti * PIECE), [ow] "s"(OPB + tj * PIECE)
            : "memory", "scc", "vcc");
        STAMP(1);
        __builtin_amdgcn_endpgm();
    } else {
        const int lane = lane_id();
        const u32 wm = wave >> 1, wn = wave & 1;
        u32 scale_bits = rfl(__builtin_bit_cast(u32, p.sx[0] * p.sw[0]));
        u32 epace = (p.dbg & 128) && ti >= 8 ? 0u : p.epace;
        asm volatile("" : "+s"(scale_bits), "+s"(epace));
        __builtin_amdgcn_s_barrier();
        u32 xa = lbase + wm * 4096u + (u32)lane * 16u;
        u32 wa = lbase + PIECE + (u32)(lane >> 5) * 1024u + wn * 512u + (u32)(lane & 31) * 16u;
        if (wave < 4 && !(p.dbg & 256)) __builtin_amdgcn_s_setprio(1);
        const u32 vdone = lbase + L_DONE + (u32)wave * 4u;
        u32 vd = lbase + L_DESC;
        const u32 mybit = wave >> 2;
        f32x16 acc[2][4];
#pragma unroll
        for (int a = 0; a < 2; ++a)
#pragma unroll
            for (int e = 0; e < 4; ++e)
#pragma unroll
                for (int r = 0; r < 16; ++r) acc[a][e][r] = 0.f;
        u32 sn, sc, sav, sk, sspin, vt, vk;
#define MF(acc, A, B) "v_mfma_f32_32x32x64_f8f6f4 " acc ", " A ", " B ", " acc " blgp:1\n\t"
#define RX0 "ds_read_b128 v[136:139], %[xa]\n\tds_read_b128 v[140:143], %[xa] offset:1024\n\t"
#define RX1 "ds_read_b128 v[144:147], %[xa] offset:2048\n\tds_read_b128 v[148:151], %[xa] offset:3072\n\t"
#define RWA(o0, o1) "ds_read_b128 v[152:155], %[wa] offset:" #o0 "\n\tds_read_b128 v[156:159], %[wa] offset:" #o1 "\n\t"
#define RWB(o0, o1) "ds_read_b128 v[160:163], %[wa] offset:" #o0 "\n\tds_read_b128 v[164:167], %[wa] offset:" #o1 "\n\t"
#define ADV "v_add_u32 %[xa], 0x8000, %[xa]\n\tv_add_u32 %[wa], 0x8000, %[wa]\n\tv_add_u32 %[vd], 4, %[vd]\n\t" \
            "v_and_b32 %[xa], 0x1ffff, %[xa]\n\tv_and_b32 %[wa], 0x1ffff, %[wa]\n\t"
#define X0 "v[136:143]"
#define X1 "v[144:151]"
#define WA "v[152:159]"
#define WB "v[160:167]"
        asm volatile(
            "s_mov_b32 %[sn], 0\n\t"
            "s_mov_b32 %[sc], 0\n\t"
            "s_mov_b32 %[sp], 0\n"
            "1:\n\t"
            "ds_read_b32 %[vt], %[dn] offset:256\n\t"
            "ds_read_b32 %[vk], %[vd]\n\t"
            "s_waitcnt lgkmcnt(0)\n\t"
            "v_readfirstlane_b32 %[sk], %[vk]\n\t"
            "v_readfirstlane_b32 %[sa], %[vt]\n\t"
            "s_cmp_eq_u32 %[sk], 0\n\t"
            "s_cbranch_scc1 8f\n\t"
            "s_bitcmp1_b32 %[sk], %[mb]\n\t"
            "s_cbranch_scc1 9f\n\t"
            "s_add_u32 %[sn], %[sn], 1\n\t"
            ADV
            "v_mov_b32 %[vt], %[sn]\n\t"
            "ds_write_b32 %[dn], %[vt]\n\t"
            "s_branch 1b\n"
            "9:\n\t"
            "s_cmp_gt_u32 %[sa], %[sn]\n\t"
            "s_cbranch_scc1 2f\n"
            "8:\n\t"
            "s_sleep 1\n\t"
            "s_add_u32 %[sp], %[sp], 1\n\t"
            "s_cmp_lt_u32 %[sp], 0x2000000\n\t"
            "s_cbranch_scc1 1b\n\t"
            "s_branch 5f\n"
            "2:\n\t"
            RWA(0, 2048) RX0 RX1 RWB(4096, 6144)
            "3:\n\t"
            "s_waitcnt lgkmcnt(4)\n\t"
            MF("%0", X0, WA)
            "s_waitcnt lgkmcnt(2)\n\t"
            MF("%4", X1, WA)
            RWA(8192, 10240)
            "ds_read_b32 %[vt], %[dn] offset:256\n\t"
            "ds_read_b32 %[vk], %[vd] offset:4\n\t"
            "s_waitcnt lgkmcnt(4)\n\t"
            MF("%1", X0, WB)
            "s_add_u32 %[sn], %[sn], 1\n\t"
            "s_add_u32 %[sc], %[sc], 1\n\t"
            MF("%5", X1, WB)
            RWB(12288, 14336)
            ADV
            "s_waitcnt lgkmcnt(2)\n\t"
            MF("%2", X0, WA)
            "v_readfirstlane_b32 %[sa], %[vt]\n\t"
            "v_readfirstlane_b32 %[sk], %[vk]\n\t"
            "v_mov_b32 %[vt], %[sn]\n\t"
            MF("%6", X1, WA)
            "s_cmp_gt_u32 %[sa], %[sn]\n\t"
            "s_cbranch_scc0 4f\n\t"
            "s_bitcmp1_b32 %[sk], %[mb]\n\t"
            "s_cbranch_scc0 4f\n\t"
            RWA(0, 2048)
            "s_waitcnt lgkmcnt(2)\n\t"
            "ds_write_b32 %[dn], %[vt]\n\t"
            MF("%3", X0, WB)
            RX0
            MF("%7", X1, WB)
            RX1 RWB(4096, 6144)
            "s_branch 3b\n"
            "4:\n\t"
            "s_waitcnt lgkmcnt(0)\n\t"
            "ds_write_b32 %[dn], %[vt]\n\t"
            MF("%3", X0, WB)
            MF("%7", X1, WB)
            "s_cmp_lt_u32 %[sc], 64\n\t"
            "s_cbranch_scc1 1b\n"
            "5:\n\t"
            "v_mov_b32 %[vt], 0x10000\n\t"
            "ds_write_b32 %[dn], %[vt]\n\t"
            "s_waitcnt lgkmcnt(0)\n\t"
            "s_nop 15\n\t"
            "s_nop 7"
            : "+v"(acc[0][0]), "+v"(acc[0][1]), "+v"(acc[0][2]), "+v"(acc[0][3]),
              "+v"(acc[1][0]), "+v"(acc[1][1]), "+v"(acc[1][2]), "+v"(acc[1][3]),
              [xa] "+v"(xa), [wa] "+v"(wa), [vd] "+v"(vd), [sn] "=&s"(sn), [sc] "=&s"(sc), [sa] "=&s"(sav), [sk] "=&s"(sk),
              [sp] "=&s"(sspin), [vt] "=&v"(vt), [vk] "=&v"(vk)
            : [dn] "v"(vdone), [mb] "s"(mybit)
            : "memory", "scc",
              "v136", "v137", "v138", "v139", "v140", "v141", "v142", "v143", "v144", "v145", "v146", "v147", "v148", "v149", "v150", "v151",
              "v152", "v153", "v154", "v155", "v156", "v157", "v158", "v159", "v160", "v161", "v162", "v163", "v164", "v165", "v166", "v167");
        const float scale = __builtin_bit_cast(float, scale_bits);
        STAMP(1);
        const bool bad = rfl(lds_rd(lbase + L_ABORT)) != 0 || sc < 64 || lbase != 0;
        const float fs = bad ? __builtin_nanf("") : scale;
        __amdgpu_buffer_rsrc_t ro = __builtin_amdgcn_make_buffer_rsrc(p.out, 0, (int)(MM * (u32)NN * 4u), 0x00020000);
        const u32 sbase = (ti * 256u + wm * 64u) * (NN * 4u) + (tj * 256u + wn * 128u) * 4u;
        const u32 ln = (xa >> 4) & 63u;
        const u32 vo = sbase + (ln >> 5) * 4u * (NN * 4u) + (ln & 31) * 16u;
#pragma unroll
        for (int a = 0; a < 2; ++a)
#pragma unroll
            for (int r = 0; r < 16; ++r) {
                u32x4 v;
#pragma unroll
                for (int e = 0; e < 4; ++e) {
                    const float y = acc[a][e][r] * fs;
                    v[e] = __builtin_bit_cast(u32, __builtin_elementwise_maximum(y, 0.f));
                }
                const u32 row = a * 32 + (r & 3) + 8 * (r >> 2);
                __builtin_amdgcn_raw_buffer_store_b128(v, ro, vo + row * (NN * 4u), 0, 16);
                if (epace) {
                    if (!(p.dbg & 128) && rfl(lds_rd(lbase + L_PDONE))) epace = 0;
                    for (u32 q = 0; q < epace; ++q) __builtin_amdgcn_s_sleep(2);
                }
            }
        __builtin_amdgcn_endpgm();
    }
}

static void build_schedule(Params& p, int q, int nburst, int early, int rot) {
    unsigned char r[2][NROUND];
    for (int l = 0; l < 2; ++l) {
        const int cs = l ? 8 - q : 8;
        int n = 0;
        for (int c = 0; c < cs; ++c) {
            r[l][n++] = (unsigned char)((c == 0 ? 0 : RD_W) | c);
            r[l][n++] = (unsigned char)((c == 0 ? RD_W : 0) | c);
        }
        for (int c = cs; c < 8; c += 2) { r[l][n++] = (unsigned char)(RD_W | c); r[l][n++] = (unsigned char)(RD_W | (c + 1)); r[l][n++] = (unsigned char)(RD_A | c); }
        for (int c = cs; c < 8; c += 2) r[l][n++] = (unsigned char)(RD_B | c);
        for (int k = (int)NROUND - (l ? nburst : 1); k < (int)NROUND; ++k) r[l][k] |= RD_BURST;
        for (int h = 0; h < 2; ++h) {
            u64 v = 0;
            for (int k = 0; k < 8; ++k) v |= (u64)r[l][h * 8 + k] << (8 * k);
            p.rounds[l][h] = v;
        }
    }
    for (int d = 0; d < NR; ++d) {
        unsigned char o[NROUND][16];
        for (int k = 0; k < (int)NROUND; ++k)
            for (int j = 0; j < 16; ++j) o[k][j] = (unsigned char)((r[d != 0][k] & RD_W) | ((r[d != 0][k] & 7) << 4) | j);
        if (d && q == 0 && early) {
            const char* tail = "X6d X7a W7d X6c W6d W7c W6a W7b W7a X7b X7c W6c X7d X6a W6b W5a X5c W5d X5d X5a W5c X6b W5b X5b";
            for (int g = 0; g < 24; ++g) {
                const char* t = tail + 4 * g;
                for (int k = 0; k < 4; ++k) {
                    const int sb = 4 * (t[2] - 'a') + k, z = 160 + 4 * g + k;
                    o[z / 16][z % 16] = (unsigned char)(((t[1] - '0') << 4) | (t[0] == 'W' ? RD_W | ((sb >> 1) + 8 * (sb & 1)) : sb));
                }
            }
        }
        for (int j = 0; j < 16; ++j)
            for (int h = 0; h < 2; ++h) {
                u64 v = 0;
                for (int k = 0; k < 8; ++k) v |= (u64)o[h * 8 + k][j] << (8 * k);
                p.sched[d][j][h] = v;
            }
    }
    p.split = 0;
    for (int c = 8 - q; c < 8; ++c)
        for (int s = 0; s < NR; ++s)
            if (s != p.rank) p.split |= 1ull << (c * 8 + s);
}

extern "C" void kernel_launch(int rank, int nranks, ncclComm_t comm,
                              hipStream_t stream,
                              void* const* d_in, const int* in_sizes, int n_in,
                              void* d_out, int out_size,
                              const int* in_dtypes, int out_dtype,
                              void* const* d_out_all,
                              void* const* d_ws_all, size_t ws_bytes) {
    (void)comm; (void)in_dtypes; (void)out_dtype; (void)d_out_all;
    if (nranks != NR || (unsigned)rank >= (unsigned)NR || n_in < 4 || ws_bytes < WS_NEED || out_size != MM * NN ||
        in_sizes[0] != MM * KS || in_sizes[1] != KS * NN) {
        fprintf(stderr, "unexpected problem shape\n");
        return;
    }
    Params p;
    p.x = (const unsigned char*)d_in[0];
    p.w = (const unsigned char*)d_in[1];
    p.sx = (const float*)d_in[2];
    p.sw = (const float*)d_in[3];
    p.out = (float*)d_out;
    for (int r = 0; r < NR; ++r) p.ws[r] = (unsigned char*)d_ws_all[(rank + r) % NR];
    p.rank = rank;
    double gbs = 58.0, gbsb = 55.0, gbs_local = 120.0, lead = 1.1, fastfrom = 55.0;
    int q = 0, nburst = 2, early = 8, rot = 1;
    p.glog = 2; p.pad = 0; p.rotlog = 0; p.epace = 0; p.slowpoll = 2; p.dbg = 0; p.qlow = 3; p.slack = 1u << 30;
    p.tau = (u32)(8192.0 * 6.4 / gbs + 0.5);
    p.taub = (u32)(8192.0 * 6.4 / gbsb + 0.5);
    p.tau_local = (u32)(8192.0 * 6.4 / gbs_local + 0.5);
    p.lead = (u32)(lead * 6400);
    p.fastfrom = (u32)(fastfrom * 100);
    build_schedule(p, q, nburst, early, rot);
    hipLaunchKernelGGL(gemm_push_kernel, dim3(256), dim3(NTHREADS), 0, stream, p);
    HIP_CHECK(hipGetLastError());
}
```

```cpp
#include <hip/hip_runtime.h>
#include <rccl/rccl.h>
#include <cstdio>
#include <cstdint>
#include <cstdlib>

#define HIP_CHECK(expr) do { hipError_t _e = (expr); if (_e != hipSuccess) { \
    fprintf(stderr, "HIP error %s at %s:%d\n", hipGetErrorString(_e), __FILE__, __LINE__); \
    return; } } while (0)

typedef unsigned int u32;
typedef unsigned long long u64;
typedef __attribute__((ext_vector_type(2))) u32 u32x2;
typedef __attribute__((ext_vector_type(4))) u32 u32x4;
typedef __attribute__((ext_vector_type(16))) float f32x16;
#define LAS __attribute__((address_space(3)))
#define GAS __attribute__((address_space(1)))

constexpr int NR = 8;
constexpr int MM = 4096, NN = 4096, KS = 512;
constexpr u32 PIECE = 16384;
constexpr u32 OPB = 2u << 20;
constexpr u32 SRCB = 4u << 20;
constexpr u32 BUFB = 32u << 20;
constexpr u32 WS_FLAGS = 0;
constexpr u32 FLAGB = 2u * 16u * 64u * 8u;
constexpr u32 WS_TMO = 128u << 10;
constexpr u32 WS_T0 = 129u << 10;
constexpr u32 WS_T1 = 130u << 10;
constexpr u32 WS_CTR = 256u << 10;
constexpr u32 WS_DATA = 2u << 20;
constexpr size_t WS_NEED = (size_t)WS_DATA + 2 * (size_t)BUFB;
constexpr u32 DEADLINE = 100000000u;

constexpr u32 NSLOT = 4;
constexpr u32 SLOTB = 2 * PIECE;
constexpr u32 L_CTRL = NSLOT * SLOTB;
constexpr u32 L_DONE = L_CTRL;
constexpr u32 L_ABORT = L_CTRL + 36;
constexpr u32 L_CALL = L_CTRL + 40;
constexpr u32 L_TR = L_CTRL + 44;
constexpr u32 L_T0 = L_CTRL + 48;
constexpr u32 L_TE = L_CTRL + 52;
constexpr u32 L_T1 = L_CTRL + 56;
constexpr u32 L_TD = L_CTRL + 60;
constexpr u32 L_P1 = L_CTRL + 64;
constexpr u32 KSPAN = 4;
constexpr u32 L_FULL = L_CTRL + 256;
constexpr u32 L_QUEUE = L_CTRL + 512;
constexpr u32 L_BYTES = L_CTRL + 1024;

constexpr int NTHREADS = 704;
constexpr u32 NROUND = 16;
constexpr int PUSHPOL = 17;
constexpr int OUTPOL = 2;
constexpr int PWIN = 1;
constexpr int NSAMP = 32;
struct Params {
    const unsigned char* x;
    const unsigned char* w;
    u32 rank;
    u32 tau0[2];
    u32 lead;
    unsigned char* ws[NR];
    u32 nsmooth;
    u32 glog;
    u32 slowpoll, fastfrom;
    u32 qlow;
    u32 ksync, margin;
    u32 fs, fo, fl, pad;
    u32 tau[2][NROUND];
    const float* sx;
    const float* sw;
    float* out;
    u64 sched[NR][16][2];
};

template <int N> __device__ __forceinline__ void waitv() { asm volatile("s_waitcnt vmcnt(%0)" ::"i"(N) : "memory"); }
__device__ __forceinline__ void waitl() { asm volatile("s_waitcnt lgkmcnt(0)" ::: "memory"); }
__device__ __forceinline__ u32 rfl(u32 v) { return __builtin_amdgcn_readfirstlane(v); }
__device__ __forceinline__ u64 rt() { return __builtin_amdgcn_s_memrealtime(); }

__device__ __forceinline__ int lane_id() {
    int l;
    asm volatile("v_mbcnt_lo_u32_b32 %0, -1, 0\n\tv_mbcnt_hi_u32_b32 %0, -1, %0" : "=v"(l));
    return l;
}
__device__ __forceinline__ u32 lds_rd(u32 addr) {
    u32 v;
    asm volatile("ds_read_b32 %0, %1\n\ts_waitcnt lgkmcnt(0)" : "=v"(v) : "v"(addr) : "memory");
    return v;
}
__device__ __forceinline__ void lds_wr(u32 addr, u32 v) {
    asm volatile("ds_write_b32 %0, %1" ::"v"(addr), "v"(v) : "memory");
}

__global__ void __launch_bounds__(NTHREADS) gemm_push_kernel(const Params p) {
    __shared__ __attribute__((aligned(16))) unsigned char smem[L_BYTES];
    const u32 lbase = (u32)(uintptr_t)(LAS unsigned char*)smem;
    const int hw_wave = rfl(threadIdx.x >> 6);
    const u32 b = blockIdx.x;
    const u32 xcd = b & 7, idx = b >> 3;
    const u32 ti = (xcd >> 1) + 4 * (idx >> 3);
    const u32 tj = (xcd & 1) * 8 + (idx & 7);
    unsigned char* myws = p.ws[0];

    if (hw_wave != 0) {
    int wave = hw_wave > 8 ? hw_wave : hw_wave - 1;
    asm volatile("" : "+s"(wave));
    if (wave == 9) {
        const u32 t0 = (u32)rt();
        const int lane = lane_id();
        if ((lane >> 1) != (int)((L_CALL - L_CTRL) / 8)) lds_wr(lbase + L_CTRL + lane * 4u, 0u);
#pragma unroll
        for (u32 q = 256; q < 1024; q += 256) lds_wr(lbase + L_CTRL + q + lane * 4u, 0u);
        waitl();
        __builtin_amdgcn_s_barrier();
        const u32 call = rfl(lds_rd(lbase + L_CALL));
        if (lane == 0) __hip_atomic_store((GAS u32*)(myws + WS_CTR + b * 512u), call, __ATOMIC_RELAXED, __HIP_MEMORY_SCOPE_AGENT);
        {
            GAS u64* tw = (GAS u64*)(myws + WS_T0);
            const u32 tr = lds_rd(lbase + L_TR);
            if (b < 8 && lane == 0) __hip_atomic_store(tw + b, ((u64)call << 32) | tr, __ATOMIC_RELAXED, __HIP_MEMORY_SCOPE_AGENT);
            for (u32 q = 0; q < 4; ++q) {
                __builtin_amdgcn_s_sleep(48);
                const u64 v = __hip_atomic_load(tw + (lane & 7), __ATOMIC_RELAXED, __HIP_MEMORY_SCOPE_AGENT);
                if (__ballot((u32)(v >> 32) == call) == ~0ull) {
                    int d = (int)((u32)v - t0);
                    d += __builtin_amdgcn_update_dpp(0, d, 0xB1, 0xf, 0xf, false);
                    d += __builtin_amdgcn_update_dpp(0, d, 0x4E, 0xf, 0xf, false);
                    d += __builtin_amdgcn_update_dpp(0, d, 0x141, 0xf, 0xf, false);
                    lds_wr(lbase + L_T0, (t0 + (u32)(d >> 3)) | 1u);
                    break;
                }
            }
        }
        __amdgpu_buffer_rsrc_t rf = __builtin_amdgcn_make_buffer_rsrc(myws + WS_FLAGS, 0, (int)FLAGB, 0x00020000);
        const u32 vfx = (ti * 64u + (u32)lane) * 8u, vfw = ((16u + tj) * 64u + (u32)lane) * 8u;
        const u32 task = ((u32)(lane & 7) * SRCB + (u32)(lane >> 3) * 16u * PIECE) | 1u;
        const u64 below = (1ull << lane) - 1ull;
        u64 seen = 0;
        u32 tail = 0;
        __builtin_amdgcn_s_setprio(2);
        const u32 sidx = (idx >> 3) * 8u + xcd;
        GAS u64* t1 = (GAS u64*)(myws + WS_T1 + (idx & 7) * 256u);
        u32 tstate = (idx & 7) ? 0u : 9u, te = 0;
        while (true) {
            if (tstate == 0) {
                te = rfl(lds_rd(lbase + L_TE));
                if (te) {
                    const u32 td = lds_rd(lbase + L_TD);
                    if (lane == 0) __hip_atomic_store(t1 + sidx, ((u64)((call << 16) | (td & 0xffffu)) << 32) | te, __ATOMIC_RELAXED, __HIP_MEMORY_SCOPE_AGENT);
                    tstate = 1;
                }
            } else if (tstate < 9 && (int)((u32)rt() - te) > (int)(200u + 100u * tstate)) {
                const u64 v = __hip_atomic_load(t1 + (lane < NSAMP ? lane : 0), __ATOMIC_RELAXED, __HIP_MEMORY_SCOPE_AGENT);
                tstate += 1;
                if (__ballot((u32)(v >> 48) == (call & 0xffffu)) == ~0ull) {
                    int d = lane < NSAMP ? (int)((u32)v - te) : 0, e = lane < NSAMP ? (int)((u32)(v >> 32) & 0xffffu) : 0;
#pragma unroll
                    for (int sh = 1; sh < 64; sh <<= 1) { d += __shfl_xor(d, sh); e += __shfl_xor(e, sh); }
                    lds_wr(lbase + L_P1, (u32)e);
                    lds_wr(lbase + L_T1, (te + (u32)(d / NSAMP)) | 1u);
                    tstate = 9;
                }
            }
            u32x2 fx, fw;
            asm volatile("buffer_load_dwordx2 %[fx], %[vfx], %[rf], 0 offen sc0 sc1\n\t"
                         "buffer_load_dwordx2 %[fw], %[vfw], %[rf], 0 offen sc0 sc1\n\t"
                         "s_waitcnt vmcnt(0)"
                         : [fx] "=&v"(fx), [fw] "=&v"(fw) : [vfx] "v"(vfx), [vfw] "v"(vfw), [rf] "s"(rf) : "memory");
            u64 fresh = __ballot((int)((fx.x - call) | (fx.y - call) | (fw.x - call) | (fw.y - call)) >= 0) & ~seen;
            u32 el = 0;
            if (!fresh) {
                el = (u32)rt() - t0;
                if (el > DEADLINE) {
                    lds_wr(lbase + L_ABORT, 1u);
                    if (lane == 0) __hip_atomic_store((GAS u32*)(myws + WS_TMO), call, __ATOMIC_RELAXED, __HIP_MEMORY_SCOPE_AGENT);
                    fresh = ~seen;
                }
            }
            if (fresh) {
                if ((fresh >> lane) & 1) {
                    const u32 at = tail + __builtin_popcountll(fresh & below);
                    lds_wr(lbase + L_QUEUE + at * 4u, task);
                }
                tail += __builtin_popcountll(fresh);
                seen |= fresh;
                if (seen == ~0ull) break;
                for (u32 q = 0; q < 4096 && tail > rfl(lds_rd(lbase + L_FULL)) + p.qlow; ++q) __builtin_amdgcn_s_sleep(2);
            } else if (el < p.fastfrom) {
                for (u32 q = 0; q < p.slowpoll; ++q) __builtin_amdgcn_s_sleep(8);
            }
        }
        waitl();
        __builtin_amdgcn_endpgm();
    } else if (wave == 10) {
        const int lane = lane_id();
        GAS u32* tab = (GAS u32*)(myws + WS_CTR + b * 512u + 256u);
        const u64 dws = (u64)p.ws[idx & 7] + WS_DATA + (u64)p.rank * SRCB + (OPB / 2 + b * 4096u);
        {
            u64 a = dws + (u64)(lane & 1) * BUFB;
            u32 dummy;
            __builtin_amdgcn_s_sleep(8);
            asm volatile("s_mov_b64 exec, 3\n\tglobal_load_dword %0, %1, off sc0 sc1\n\ts_mov_b64 exec, -1\n\ts_barrier\n\ts_waitcnt vmcnt(0)" : "=&v"(dummy) : "v"(a) : "memory");
        }
        asm volatile("" ::: "memory");
        tab[lane] = (u32)lane + 1u;
        const u32 call = rfl(lds_rd(lbase + L_CALL));
        waitv<0>();
        __amdgpu_buffer_rsrc_t rtag = __builtin_amdgcn_make_buffer_rsrc((void*)tab, 0, 256, 0x00020000);
        __amdgpu_buffer_rsrc_t rs = __builtin_amdgcn_make_buffer_rsrc(myws + WS_DATA + (size_t)(call & 1) * BUFB, 0, (int)BUFB, 0x00020000);
        {
            __builtin_amdgcn_s_sleep(16);
            u64 a = (u64)myws + WS_DATA + (u64)(call & 1) * BUFB + (u64)(lane & 15) * OPB + (OPB / 2 + b * 4096u);
            if (lane >= 16) a = (u64)p.out + ((u64)ti * 256u + (u64)(lane & 3) * 85u) * (NN * 4u) + tj * 1024u;
            u32 dummy;
            asm volatile("s_mov_b64 exec, 0xfffff\n\tglobal_load_dword %0, %1, off sc0 sc1\n\ts_mov_b64 exec, -1\n\ts_waitcnt vmcnt(0)" : "=v"(dummy) : "v"(a) : "memory");
        }
        const u32 vo = (u32)lane * 16u, vz = 0u;
        const u32 vdn = lbase + L_DONE + (u32)(lane & 7) * 4u;
        u32 vq = lbase + L_QUEUE, vt, vd;
        u32 si, sq, sx, sw, sl, sp;
        __builtin_amdgcn_s_setprio(2);
#define DMA4_S(so) "buffer_load_dwordx4 %[vo], %[rs], " so " offen sc0 sc1 lds\n\t" \
                   "buffer_load_dwordx4 %[vo], %[rs], " so " offen offset:1024 sc0 sc1 lds\n\t" \
                   "buffer_load_dwordx4 %[vo], %[rs], " so " offen offset:2048 sc0 sc1 lds\n\t" \
                   "buffer_load_dwordx4 %[vo], %[rs], " so " offen offset:3072 sc0 sc1 lds\n\t" \
                   "s_add_u32 m0, m0, 0x1000\n\ts_add_u32 " so ", " so ", 0x1000\n\t"
        asm volatile(
            "s_mov_b32 %[si], 0\n\t"
            "s_mov_b32 %[sp], 0\n\t"
            "s_mov_b32 %[sl], %[lb]\n"
            "1:\n\t"
            "ds_read_b32 %[vt], %[vq]\n\t"
            "ds_read_b32 %[vd], %[vdn]\n\t"
            "s_cmp_eq_u32 %[si], 256\n\t"
            "s_cbranch_scc1 9f\n\t"
            "s_waitcnt lgkmcnt(0)\n\t"
            "v_readfirstlane_b32 %[sq], %[vt]\n\t"
            "v_lshl_add_u32 %[vd], %[vd], 2, 16\n\t"
            "v_cmp_lt_u32 vcc, %[si], %[vd]\n\t"
            "s_cmp_eq_u64 vcc, -1\n\t"
            "s_cselect_b32 %[sq], %[sq], 0\n\t"
            "s_cmp_lg_u32 %[sq], 0\n\t"
            "s_cbranch_scc1 2f\n\t"
            "s_sleep 1\n\t"
            "s_add_u32 %[sp], %[sp], 1\n\t"
            "s_cmp_lt_u32 %[sp], 0x2000000\n\t"
            "s_cbranch_scc1 1b\n\t"
            "s_branch 9f\n"
            "2:\n\t"
            "s_mov_b32 m0, %[sl]\n\t"
            "s_add_u32 %[sx], %[sq], %[ox]\n\t"
            "s_add_u32 %[sw], %[sq], %[ow]\n\t"
            DMA4_S("%[sx]") DMA4_S("%[sx]") DMA4_S("%[sx]") DMA4_S("%[sx]")
            DMA4_S("%[sw]") DMA4_S("%[sw]")
            "ds_read_b32 %[vt], %[vq] offset:4\n\t"
            "ds_read_b32 %[vd], %[vdn]\n\t"
            "v_add_u32 %[vq], 4, %[vq]\n\t"
            "s_add_u32 %[sl], %[sl], 0x8000\n\t"
            "s_and_b32 %[sl], %[sl], 0x1ffff\n\t"
            DMA4_S("%[sw]")
            "s_waitcnt lgkmcnt(0)\n\t"
            "v_readfirstlane_b32 %[sq], %[vt]\n\t"
            "v_lshl_add_u32 %[vd], %[vd], 2, 12\n\t"
            "v_cmp_lt_u32 vcc, %[si], %[vd]\n\t"
            DMA4_S("%[sw]")
            "s_mov_b32 m0, %[lf]\n\t"
            "s_cmp_eq_u64 vcc, -1\n\t"
            "buffer_load_dword %[vz], %[rt], %[si] offen lds\n\t"
            "s_cselect_b32 %[sq], %[sq], 0\n\t"
            "s_add_u32 %[si], %[si], 4\n\t"
            "s_cmp_lg_u32 %[sq], 0\n\t"
            "s_cbranch_scc1 2b\n\t"
            "s_branch 1b\n"
            "9:\n\t"
            "s_waitcnt vmcnt(0) lgkmcnt(0)"
            : [vq] "+v"(vq), [vt] "=&v"(vt), [vd] "=&v"(vd), [si] "=&s"(si), [sq] "=&s"(sq), [sx] "=&s"(sx), [sw] "=&s"(sw),
              [sl] "=&s"(sl), [sp] "=&s"(sp)
            : [vdn] "v"(vdn), [vo] "v"(vo), [vz] "v"(vz), [rs] "s"(rs), [rt] "s"(rtag), [lb] "s"(lbase), [lf] "s"(lbase + L_FULL),
              [ox] "s"(ti * PIECE - 1u), [ow] "s"(OPB + tj * PIECE - 1u)
            : "memory", "scc", "vcc");
        __builtin_amdgcn_endpgm();
    } else {
        const int lane = lane_id();
        const u32 wm = wave >> 1, wn = wave & 1;
        __builtin_amdgcn_s_barrier();
        asm volatile("" ::: "memory");
        u32 scale_bits = rfl(__builtin_bit_cast(u32, p.sx[0] * p.sw[0]));
        u64 outp = (u64)p.out;
        asm volatile("" : "+s"(scale_bits), "+s"(outp));
        u32 xa = lbase + wm * 4096u + (u32)lane * 16u;
        u32 wa = lbase + PIECE + (u32)(lane >> 5) * 1024u + wn * 512u + (u32)(lane & 31) * 16u;
        if (wave < 4) __builtin_amdgcn_s_setprio(1);
        const u32 vdone = lbase + L_DONE + (u32)wave * 4u;
        f32x16 acc[2][4];
#pragma unroll
        for (int a = 0; a < 2; ++a)
#pragma unroll
            for (int e = 0; e < 4; ++e)
#pragma unroll
                for (int r = 0; r < 16; ++r) acc[a][e][r] = 0.f;
        u32 sn, sav, sspin, vt;
#define MF(acc, A, B) "v_mfma_f32_32x32x64_f8f6f4 " acc ", " A ", " B ", " acc " blgp:1\n\t"
#define RX0 "ds_read_b128 v[136:139], %[xa]\n\tds_read_b128 v[140:143], %[xa] offset:1024\n\t"
#define RX1 "ds_read_b128 v[144:147], %[xa] offset:2048\n\tds_read_b128 v[148:151], %[xa] offset:3072\n\t"
#define RWA(o0, o1) "ds_read_b128 v[152:155], %[wa] offset:" #o0 "\n\tds_read_b128 v[156:159], %[wa] offset:" #o1 "\n\t"
#define RWB(o0, o1) "ds_read_b128 v[160:163], %[wa] offset:" #o0 "\n\tds_read_b128 v[164:167], %[wa] offset:" #o1 "\n\t"
#define X0 "v[136:143]"
#define X1 "v[144:151]"
#define WA "v[152:159]"
#define WB "v[160:167]"
        asm volatile(
            "s_mov_b32 %[sn], 0\n\t"
            "s_mov_b32 %[sp], 0\n"
            "1:\n\t"
            "ds_read_b32 %[vt], %[dn] offset:256\n\t"
            "s_waitcnt lgkmcnt(0)\n\t"
            "v_readfirstlane_b32 %[sa], %[vt]\n\t"
            "s_cmp_gt_u32 %[sa], %[sn]\n\t"
            "s_cbranch_scc1 2f\n\t"
            "s_sleep 1\n\t"
            "s_add_u32 %[sp], %[sp], 1\n\t"
            "s_cmp_lt_u32 %[sp], 0x2000000\n\t"
            "s_cbranch_scc1 1b\n\t"
            "s_branch 5f\n"
            "2:\n\t"
            RWA(0, 2048) RX0 RX1 RWB(4096, 6144)
            "3:\n\t"
            "s_waitcnt lgkmcnt(4)\n\t"
            MF("%0", X0, WA)
            "s_waitcnt lgkmcnt(2)\n\t"
            MF("%4", X1, WA)
            RWA(8192, 10240)
            "ds_read_b32 %[vt], %[dn] offset:256\n\t"
            "s_waitcnt lgkmcnt(3)\n\t"
            MF("%1", X0, WB)
            "s_add_u32 %[sn], %[sn], 1\n\t"
            MF("%5", X1, WB)
            RWB(12288, 14336)
            "v_add_u32 %[xa], 0x8000, %[xa]\n\t"
            "v_add_u32 %[wa], 0x8000, %[wa]\n\t"
            "v_and_b32 %[xa], 0x1ffff, %[xa]\n\t"
            "v_and_b32 %[wa], 0x1ffff, %[wa]\n\t"
            "s_waitcnt lgkmcnt(2)\n\t"
            MF("%2", X0, WA)
            "v_readfirstlane_b32 %[sa], %[vt]\n\t"
            "v_mov_b32 %[vt], %[sn]\n\t"
            MF("%6", X1, WA)
            "s_cmp_gt_u32 %[sa], %[sn]\n\t"
            "s_cbranch_scc0 4f\n\t"
            RWA(0, 2048)
            "s_sub_u32 %[sa], %[sa], %[sn]\n\t"
            "s_cmp_ge_u32 %[sn], %[pe]\n\t"
            "s_cselect_b32 %[sa], %[pa], %[sa]\n\t"
            "s_cmp_ge_u32 %[sa], 2\n\t"
            "s_cbranch_scc1 6f\n\t"
            "s_setprio 0\n\t"
            "s_branch 7f\n"
            "6:\n\ts_setprio 1\n"
            "7:\n\t"
            "s_waitcnt lgkmcnt(2)\n\t"
            "ds_write_b32 %[dn], %[vt]\n\t"
            MF("%3", X0, WB)
            RX0
            MF("%7", X1, WB)
            RX1 RWB(4096, 6144)
            "s_branch 3b\n"
            "4:\n\t"
            "s_waitcnt lgkmcnt(0)\n\t"
            "ds_write_b32 %[dn], %[vt]\n\t"
            MF("%3", X0, WB)
            MF("%7", X1, WB)
            "s_cmp_lt_u32 %[sn], 64\n\t"
            "s_cbranch_scc1 1b\n"
            "5:\n\t"
            "ds_read_b32 %[vt], %[ab]\n\t"
            "s_waitcnt lgkmcnt(0)\n\t"
            "s_nop 15\n\t"
            "s_nop 7"
            : "+v"(acc[0][0]), "+v"(acc[0][1]), "+v"(acc[0][2]), "+v"(acc[0][3]),
              "+v"(acc[1][0]), "+v"(acc[1][1]), "+v"(acc[1][2]), "+v"(acc[1][3]),
              [xa] "+v"(xa), [wa] "+v"(wa), [sn] "=&s"(sn), [sa] "=&s"(sav), [sp] "=&s"(sspin), [vt] "=&v"(vt)
            : [dn] "v"(vdone), [ab] "v"(lbase + L_ABORT), [pe] "s"(60u), [pa] "s"(wave < 4 ? 9u : 0u)
            : "memory", "scc",
              "v136", "v137", "v138", "v139", "v140", "v141", "v142", "v143", "v144", "v145", "v146", "v147", "v148", "v149", "v150", "v151",
              "v152", "v153", "v154", "v155", "v156", "v157", "v158", "v159", "v160", "v161", "v162", "v163", "v164", "v165", "v166", "v167");
        const bool bad = rfl(vt) != 0 || sn < 64 || lbase != 0;
        const float fs = bad ? __builtin_nanf("") : __builtin_bit_cast(float, scale_bits);
        __amdgpu_buffer_rsrc_t ro = __builtin_amdgcn_make_buffer_rsrc((void*)outp, 0, (int)(MM * (u32)NN * 4u), 0x00020000);
        const u32 sbase = (ti * 256u + wm * 64u) * (NN * 4u) + (tj * 256u + wn * 128u) * 4u;
        const u32 ln = (xa >> 4) & 63u;
        const u32 vo = sbase + (ln >> 5) * 4u * (NN * 4u) + (ln & 31) * 16u;
#pragma unroll
        for (int a = 0; a < 2; ++a)
#pragma unroll
            for (int r = 0; r < 16; ++r) {
                u32x4 v;
#pragma unroll
                for (int e = 0; e < 4; ++e) {
                    const float y = acc[a][e][r] * fs;
                    v[e] = __builtin_bit_cast(u32, __builtin_elementwise_maximum(y, 0.f));
                }
                const u32 row = a * 32 + (r & 3) + 8 * (r >> 2);
                __builtin_amdgcn_raw_buffer_store_b128(v, ro, vo + row * (NN * 4u), 0, OUTPOL);
            }
        __builtin_amdgcn_endpgm();
    }
    }
    {
        __builtin_amdgcn_s_setprio(3);
        const int lane = lane_id();
        const u32 dofs = idx & 7;
        const u32 m = (idx >> 3) * 8 + xcd;
        const u32 pos = (m & 24u) | ((m + dofs) & 7u);
        __amdgpu_buffer_rsrc_t rx = __builtin_amdgcn_make_buffer_rsrc((void*)p.x, 0, (int)OPB, 0x00020000);
        __amdgpu_buffer_rsrc_t rw = __builtin_amdgcn_make_buffer_rsrc((void*)p.w, 0, (int)OPB, 0x00020000);
        const u32 vx = (u32)(lane & 31) * 512u + (u32)(lane >> 5) * 16u;
        const u32 vw = (u32)lane * 4u;
        u32x4 o[8], o2[8];
        u32 vo, vs, vo2, vs2;
        auto load_w = [&](u32 u, u32 (&cur)[32]) {
            const u32 so = (((u >> 5) & 7) * 64u + (u & 1) * 32u) * 4096u + ((u >> 1) & 15) * 256u;
#pragma unroll
            for (int k = 0; k < 32; ++k) cur[k] = __builtin_amdgcn_raw_buffer_load_b32(rw, vw, so + k * 4096u, 0);
        };
        auto conv_w = [&](u32 u, const u32 (&cur)[32]) {
#pragma unroll
            for (int t = 0; t < 8; ++t) {
                const u32 d0 = cur[4 * t], d1 = cur[4 * t + 1], d2 = cur[4 * t + 2], d3 = cur[4 * t + 3];
                const u32 lo01 = __builtin_amdgcn_perm(d1, d0, 0x05010400u), hi01 = __builtin_amdgcn_perm(d1, d0, 0x07030602u);
                const u32 lo23 = __builtin_amdgcn_perm(d3, d2, 0x05010400u), hi23 = __builtin_amdgcn_perm(d3, d2, 0x07030602u);
                const int g = t >> 2;
                o2[0 + g][t & 3] = __builtin_amdgcn_perm(lo23, lo01, 0x05040100u);
                o2[2 + g][t & 3] = __builtin_amdgcn_perm(lo23, lo01, 0x07060302u);
                o2[4 + g][t & 3] = __builtin_amdgcn_perm(hi23, hi01, 0x05040100u);
                o2[6 + g][t & 3] = __builtin_amdgcn_perm(hi23, hi01, 0x07060302u);
            }
            vo2 = (u >> 1) * PIECE + (u & 1) * 2048u; vs2 = 4096u;
#pragma unroll
            for (int i = 0; i < 8; ++i) asm volatile("" : "+v"(o2[i]));
        };
        auto load_x = [&](u32 u, u32x4 (&cx)[8]) {
            const u32 so = (u & 31) * (128u * 512u) + (u >> 5) * 64u;
#pragma unroll
            for (int i = 0; i < 8; ++i) cx[i] = __builtin_amdgcn_raw_buffer_load_b128(rx, vx, so + (i >> 1) * 32u * 512u + (i & 1) * 32u, 0);
        };
        auto conv_x = [&](u32 u, const u32x4 (&cx)[8]) {
            vo2 = u * 8192u; vs2 = 2048u;
#pragma unroll
            for (int i = 0; i < 8; ++i) { o2[i] = cx[i]; asm volatile("" : "+v"(o2[i])); }
        };
        auto rotate = [&]() {
#pragma unroll
            for (int i = 0; i < 8; ++i) o[i] = o2[i];
            vo = vo2; vs = vs2;
        };
        u32 un = pos;
        asm volatile("" : "+s"(rx));
        u32 cprev = *(const __attribute__((address_space(4))) u32*)(u64)(myws + WS_CTR + b * 512u);
        u64 dws = (u64)p.ws[dofs];
        __builtin_amdgcn_sched_barrier(0);
        u32x4 cx0[8];
        load_x(un, cx0);
        __builtin_amdgcn_sched_barrier(0);
        asm volatile("" ::: "memory");
        u32 tau = dofs ? p.tau0[1] : p.tau0[0], pre = p.lead, rank = p.rank;
        u64 dbase = dws + WS_DATA + (u64)rank * SRCB;
        const u32* taus = p.tau[dofs != 0];
        const u32 lane16 = (u32)lane * 16u, lanef = (u32)lane * FLAGB;
        u32 ph0 = (pos * tau) >> 2;
        asm volatile("" : "+s"(dws), "+s"(dbase), "+s"(tau), "+s"(pre), "+s"(rank), "+s"(ph0), "+s"(cprev));
        const u32 call = cprev + 1u;
        __amdgpu_buffer_rsrc_t rd = __builtin_amdgcn_make_buffer_rsrc((void*)(dbase + (u64)(call & 1) * BUFB), 0, (int)SRCB, 0x00020000);
        __amdgpu_buffer_rsrc_t rf = __builtin_amdgcn_make_buffer_rsrc((void*)(dws + WS_FLAGS), 0, (int)FLAGB, 0x00020000);
        lds_wr(lbase + L_CALL, call);
        vo = un * 8192u; vs = 2048u;
        u32 va0 = lane16 + vo;
        asm volatile("" : "+s"(rd));
        asm volatile("" : "+v"(va0));
        asm volatile("" : "+v"(cx0[0]), "+v"(cx0[1]));
#pragma unroll
        for (int i = 0; i < 8; ++i) o[i] = cx0[i];
        const bool now = ph0 <= pre;
        if (now) {
            __builtin_amdgcn_raw_buffer_store_b128(o[0], rd, va0, 0, PUSHPOL);
            __builtin_amdgcn_raw_buffer_store_b128(o[1], rd, va0 + 1024u, 0, PUSHPOL);
        }
        asm volatile("" ::: "memory");
        const u32 tready = (u32)rt();
        u32 round = tready * 64u - pre;
        const u32 sb0 = round + ph0;
        lds_wr(lbase + L_TR, tready);
        u32 tk0 = 0;
        u32 pflag = FLAGB;
#define FLAG_OF(u) ((((((u) >> 8) * 16u + (((u) >> 1) & 15)) * 64u + (((u) >> 5) & 7) * 8u + rank) * 2u + ((u) & 1)) * 4u)
#define PACE(t) do { const u32 due_ = (t); while ((int)((u32)rt() * 64u - due_) < 0) __builtin_amdgcn_s_sleep(1); } while (0)
#define PST(i) do { __builtin_amdgcn_raw_buffer_store_b128(o[i], rd, lane16 + vo + ((i) >> 1) * vs + ((i) & 1) * 1024u, 0, PUSHPOL); asm volatile("" ::: "memory"); } while (0)
#define PFL(f) do { __builtin_amdgcn_raw_buffer_store_b32(call, rf, (f) + lanef, 0, 17); asm volatile("" ::: "memory"); } while (0)
#define PUSH_SMOOTH(SB, MID, NL, LOADS, CONV) do { \
            const u32 sbase = (SB); \
            if (k) { PACE(sbase); waitv<PWIN>(); PST(0); PST(1); } else if (!now) { PACE(sbase); PST(0); PST(1); } \
            MID; \
            if (k == p.ksync - KSPAN) tk0 = (u32)rt(); \
            if (k == p.ksync) { const u32 t_ = (u32)rt(); lds_wr(lbase + L_TD, t_ - tk0); lds_wr(lbase + L_TE, (t_ - ((sbase - round) >> 6)) | 1u); } \
            PACE(sbase + 8u * tau); waitv<PWIN>(); \
            PFL(pflag);          \
            PST(2); PST(3); \
            LOADS; \
            asm volatile("" ::: "memory"); \
            PACE(sbase + 16u * tau); waitv<PWIN + NL>(); PST(4); PST(5); \
            PACE(sbase + 24u * tau); waitv<PWIN>(); PST(6); PST(7); \
            CONV; \
            pflag = FLAG_OF(un); \
        } while (0)
#define PUSH_ORDERED(LOADS, CONV) do { \
            if (pflag != FLAGB) { waitv<0>(); PFL(pflag); pflag = FLAGB; } \
            const u32 gb = round + ((((pos >> p.glog) << (p.glog + 2)) + (pos & ((1u << p.glog) - 1u))) * tau >> 2); \
            const u32 gs = (tau << p.glog) >> 2; \
            PACE(gb); \
            PST(0); PST(1); PACE(gb + gs); PST(2); PST(3); PACE(gb + 2u * gs); PST(4); PST(5); PACE(gb + 3u * gs); PST(6); PST(7); \
            waitv<0>(); \
            PFL(FLAG_OF(un)); \
            LOADS; \
            CONV; \
        } while (0)
        {
            const u32 k = 0, nu = 256u | pos; (void)k;
            u32 cur[32];
            PUSH_SMOOTH(sb0, (waitl(), __builtin_amdgcn_s_barrier()), 32, load_w(nu, cur), conv_w(nu, cur));
            rotate();
            un = nu;
        }
        const u32 nsmooth = dofs ? p.nsmooth : NROUND - 1;
        u32 te = 1, ts = dofs ? 1u : 0u, tscale = 0;
        u64 sch = p.sched[dofs][pos >> 1][0] >> 8;
#pragma nounroll
        for (u32 k = 1; k + 1 < NROUND; ++k) {
            round += 32u * tau;
            tau = taus[k];
            if (te) {
                const u32 tz = rfl(lds_rd(lbase + L_T0));
                if (tz) {
                    const int dl = (int)(tz - tready);
                    if (dl > -800 && dl < 800) round += (u32)dl * 64u;
                    te = 0;
                }
            }
            if (k > p.ksync && ts) {
                const u32 tz = rfl(lds_rd(lbase + L_T1));
                if (tz) {
                    const u32 per = rfl(lds_rd(lbase + L_P1)) * 64u / ((u32)NSAMP * KSPAN);
                    const u32 nr = tz * 64u + (k - p.ksync) * per + p.margin;
                    if (per > 24u * 900u && per < 48u * 900u && nr - round + 128000u < 256000u) {
                        round = nr;
                        tscale = per;
                    }
                    ts = 0;
                }
            }
            if (tscale) tau = (tscale * (k < nsmooth ? p.fs : p.fo)) >> 15;
            sch >>= 8;
            if (k == 7) sch = p.sched[dofs][pos >> 1][1];
            const u32 nu = (((u32)sch & 255u) << 1) | (pos & 1);
            if (k < nsmooth) {
                if (nu & 256) { u32 cur[32]; PUSH_SMOOTH(round + ((pos * tau) >> 2), (void)0, 32, load_w(nu, cur), conv_w(nu, cur)); }
                else { u32x4 cx[8]; PUSH_SMOOTH(round + ((pos * tau) >> 2), (void)0, 8, load_x(nu, cx), conv_x(nu, cx)); }
            } else {
                if (nu & 256) { u32 cur[32]; PUSH_ORDERED(load_w(nu, cur), conv_w(nu, cur)); }
                else { u32x4 cx[8]; PUSH_ORDERED(load_x(nu, cx), conv_x(nu, cx)); }
            }
            rotate();
            un = nu;
        }
        round += 32u * tau;
        tau = tscale ? (tscale * p.fl) >> 15 : taus[NROUND - 1];
        PUSH_ORDERED((void)0, (void)0);
        waitv<0>();
        __builtin_amdgcn_endpgm();
    }
}

static void build_schedule(Params& p) {
    for (int d = 0; d < NR; ++d) {
        unsigned char o[256];
        for (int z = 0; z < 256; ++z) {
            const int c = z >> 5, second = (z >> 4) & 1;
            o[z] = (unsigned char)((((c == 0) == (second != 0)) ? 128 : 0) | (c << 4) | (z & 15));
        }
        if (d) {
            const char* tail = (d & 1) ? "W4d W4c X4d X4c W6c W7b X7c W7c W4a X7d X7a X4a W7a X4b W7d W4b "
                                         "X6d W6a X6c W6d X5d W5d X7b W5c W5a X6a X5b X5c X5a W6b X6b W5b"
                                       : "W4d X7c X7a X4c W7a W7b W4a X7b X7d W7d W4c X4d X4a W5a W4b W7c "
                                         "X5d X4b W6a W5d X5c W5c X6c X5a X6d W6d W6c X5b W5b X6a X6b W6b";
            for (int g = 0; g < 32; ++g) {
                const char* t = tail + 4 * g;
                for (int k = 0; k < 4; ++k) {
                    const int sb = 4 * (t[2] - 'a') + k;
                    o[128 + 4 * g + k] = (unsigned char)(((t[1] - '0') << 4) | (t[0] == 'W' ? 128 | ((sb >> 1) + 8 * (sb & 1)) : sb));
                }
            }
        }
        for (int j = 0; j < 16; ++j)
            for (int h = 0; h < 2; ++h) {
                u64 v = 0;
                for (int k = 0; k < 8; ++k) v |= (u64)o[(h * 8 + k) * 16 + j] << (8 * k);
                p.sched[d][j][h] = v;
            }
    }
}

extern "C" void kernel_launch(int rank, int nranks, ncclComm_t comm,
                              hipStream_t stream,
                              void* const* d_in, const int* in_sizes, int n_in,
                              void* d_out, int out_size,
                              const int* in_dtypes, int out_dtype,
                              void* const* d_out_all,
                              void* const* d_ws_all, size_t ws_bytes) {
    (void)comm; (void)in_dtypes; (void)out_dtype; (void)d_out_all;
    if (nranks != NR || (unsigned)rank >= (unsigned)NR || n_in < 4 || ws_bytes < WS_NEED || out_size != MM * NN ||
        in_sizes[0] != MM * KS || in_sizes[1] != KS * NN) {
        fprintf(stderr, "unexpected problem shape\n");
        return;
    }
    Params p;
    p.x = (const unsigned char*)d_in[0];
    p.w = (const unsigned char*)d_in[1];
    p.sx = (const float*)d_in[2];
    p.sw = (const float*)d_in[3];
    p.out = (float*)d_out;
    for (int r = 0; r < NR; ++r) p.ws[r] = (unsigned char*)d_ws_all[(rank + r) % NR];
    p.rank = (u32)rank;
    double gbs = 58.8, gbsb = 55.0, gbs_local = 200.0, lead = 0.4, fastfrom = 60.0;
    int nord = 2;
    const double gbsf = 59.5;
    const u32 nfast = 3;
    p.ksync = 9;
    double margin = -0.2, fs = 1.01, fo = 1.07, fl = 1.07;
    p.glog = 2; p.slowpoll = 5; p.qlow = 2;
    p.nsmooth = NROUND - nord;
    for (u32 k = 0; k < NROUND; ++k) {
        double g = k < nfast ? gbsf : k < p.nsmooth ? gbs : gbsb;
        p.tau[0][k] = (u32)(8192.0 * 6.4 / gbs_local + 0.5);
        p.tau[1][k] = (u32)(8192.0 * 6.4 / g + 0.5);
    }
    p.tau0[0] = p.tau[0][0]; p.tau0[1] = p.tau[1][0];
    p.lead = (u32)(lead * 6400);
    p.margin = (u32)(int)(margin * 6400); p.fs = (u32)(fs * 1024); p.fo = (u32)(fo * 1024); p.fl = (u32)(fl * 1024); p.pad = 0;
    p.fastfrom = (u32)(fastfrom * 100);
    build_schedule(p);
    hipLaunchKernelGGL(gemm_push_kernel, dim3(256), dim3(NTHREADS), 0, stream, p);
    HIP_CHECK(hipGetLastError());
}
```
